```python
import numpy as np

kernel_name = "gpu_gemm_ar_m4096_k4096_n4096_f32_q8"


ABSMAX_RTOL = 0.02

COMPUTE_DTYPE = "float32"
EPILOGUE = "quant_int8"
IN_QUANT = None
OUT_QUANT = "int8"
IN_DTYPES = {"x": "float32", "w_mat": "float32"}
WORLD_SIZE = 8

def setup_inputs():

    x = np.random.randn(4096, 4096).astype(np.float32)
    w_mat = (np.random.randn(4096, 4096) / np.sqrt(4096)).astype(np.float32)
    return {"x": x, "w_mat": w_mat}

def _quant_dequant(y, amax):
    scale = amax / 127.0
    q = np.clip(np.round(y / scale), -127.0, 127.0)
    return (q * scale).astype(np.float32)
def _epilogue(y):
    return _quant_dequant(y, np.abs(y).max())

def reference(x, w_mat):
    y = (x @ w_mat).astype(np.float32)
    return _epilogue(y)
```

```cpp
#include <hip/hip_runtime.h>
#include <rccl/rccl.h>
#include <cstdio>

#define GAS __attribute__((address_space(1)))
#define LAS __attribute__((address_space(3)))
typedef unsigned u32x2 __attribute__((ext_vector_type(2)));
typedef unsigned u32x4 __attribute__((ext_vector_type(4)));
typedef float f32x2 __attribute__((ext_vector_type(2)));
typedef float f32x4 __attribute__((ext_vector_type(4)));
typedef float f32x16 __attribute__((ext_vector_type(16)));
typedef __bf16 bf16x2 __attribute__((ext_vector_type(2)));
typedef __bf16 bf16x8 __attribute__((ext_vector_type(8)));
typedef unsigned long long u64;

namespace {

constexpr int NR = 8;
constexpr int MM = 4096, NN = 4096, KS = 512;
constexpr int NWG = 256, NTHR = 512, NGEMM = 128;
#define W1 18
#define W2 18
constexpr u64 DEADLINE = 50000000ull;

constexpr unsigned OFF_TMO = 0;
constexpr unsigned OFF_CTR = 4096;
constexpr unsigned OFF_AMAX = 16384;
constexpr unsigned OFF_LOCX = 32768;
constexpr unsigned OFF_LOCW = 49152;
constexpr unsigned OFF_EXPF = 65536;
constexpr unsigned OFF_OUTF = 131072;
constexpr unsigned OFF_DBG = 262144;
constexpr unsigned OFF_EXPX = 1u << 20;
constexpr unsigned OFF_EXPW = OFF_EXPX + (4u << 20);
constexpr unsigned OFF_EXPO = OFF_EXPW + (4u << 20);
constexpr unsigned OFF_INX = 16u << 20;
constexpr unsigned OFF_INW = 32u << 20;
constexpr size_t WS_NEED = 40u << 20;

constexpr int NS = 4;
constexpr unsigned L_CTRL = NS * 32768;
constexpr unsigned L_ORDER = L_CTRL;
constexpr unsigned L_CNT = L_CTRL + 256;
constexpr unsigned L_FULL = L_CTRL + 272;
constexpr unsigned L_CONS = L_CTRL + 288;
constexpr unsigned L_WMAX = L_CTRL + 304;
constexpr unsigned L_GMAX = L_CTRL + 320;
constexpr unsigned L_FAIL = L_CTRL + 336;
constexpr unsigned LDS_BYTES = L_CTRL + 512;

struct Params {
  const float* x;
  const float* w;
  char* out;
  char* ws;
  char* ws_all[NR];
  int rank;
  int pad;
};

__device__ __forceinline__ u64 rt() { return __builtin_amdgcn_s_memrealtime(); }
__device__ __forceinline__ unsigned rfl(unsigned v) { return __builtin_amdgcn_readfirstlane(v); }
__device__ __forceinline__ unsigned rdl(unsigned v, unsigned l) { return __builtin_amdgcn_readlane(v, l); }
__device__ __forceinline__ char* mkptr(unsigned lo, unsigned hi) { return (char*)(((u64)hi << 32) | lo); }

#define WAITV(n) asm volatile("s_waitcnt vmcnt(" #n ")" ::: "memory")
#define WAITL() asm volatile("s_waitcnt lgkmcnt(0)" ::: "memory")

template <int IMM>
__device__ __forceinline__ void st16(char* sbase, unsigned voff, u32x4 v) {
  asm volatile("s_nop 4\n\tglobal_store_dwordx4 %0, %1, %2 offset:%3 sc0 sc1\n\ts_nop 1"
               :: "v"(voff), "v"(v), "s"(sbase), "n"(IMM) : "memory");
}
__device__ __forceinline__ void st8v(char* p, u32x2 v) {
  asm volatile("global_store_dwordx2 %0, %1, off sc0 sc1\n\ts_nop 1" :: "v"(p), "v"(v) : "memory");
}
__device__ __forceinline__ void st4v(char* p, unsigned v) {
  asm volatile("global_store_dword %0, %1, off sc0 sc1\n\ts_nop 1" :: "v"(p), "v"(v) : "memory");
}
__device__ __forceinline__ unsigned lds_r32(unsigned a) {
  unsigned v;
  asm volatile("ds_read_b32 %0, %1\n\ts_waitcnt lgkmcnt(0)" : "=v"(v) : "v"(a) : "memory");
  return v;
}
__device__ __forceinline__ u32x2 lds_r64(unsigned a) {
  u32x2 v;
  asm volatile("ds_read_b64 %0, %1\n\ts_waitcnt lgkmcnt(0)" : "=v"(v) : "v"(a) : "memory");
  return v;
}
__device__ __forceinline__ void lds_w32(unsigned a, unsigned v) {
  asm volatile("ds_write_b32 %0, %1" :: "v"(a), "v"(v) : "memory");
}
__device__ __forceinline__ void lds_w64(unsigned a, u32x2 v) {
  asm volatile("ds_write_b64 %0, %1" :: "v"(a), "v"(v) : "memory");
}
__device__ __forceinline__ unsigned umax(unsigned a, unsigned b) { return a > b ? a : b; }
__device__ __forceinline__ unsigned wave_max(unsigned v) {
  v = umax(v, (unsigned)__builtin_amdgcn_update_dpp(0, (int)v, 0x111, 0xf, 0xf, true));
  v = umax(v, (unsigned)__builtin_amdgcn_update_dpp(0, (int)v, 0x112, 0xf, 0xf, true));
  v = umax(v, (unsigned)__builtin_amdgcn_update_dpp(0, (int)v, 0x114, 0xf, 0xf, true));
  v = umax(v, (unsigned)__builtin_amdgcn_update_dpp(0, (int)v, 0x118, 0xf, 0xf, true));
  v = umax(v, (unsigned)__builtin_amdgcn_update_dpp(0, (int)v, 0x142, 0xa, 0xf, false));
  v = umax(v, (unsigned)__builtin_amdgcn_update_dpp(0, (int)v, 0x143, 0xc, 0xf, false));
  return rdl(v, 63);
}
__device__ __forceinline__ unsigned f2u(float f) { return __builtin_bit_cast(unsigned, f); }
__device__ __forceinline__ float u2f(unsigned u) { return __builtin_bit_cast(float, u); }
__device__ __forceinline__ unsigned pk_bf16(float a, float b) {
  f32x2 f = {a, b};
  return __builtin_bit_cast(unsigned, __builtin_convertvector(f, bf16x2));
}

#define STAMP(k) do { } while (0)

__device__ __forceinline__ bool give_up(char* ws, unsigned tmo, unsigned call, u64 t0) {
  if (tmo == call) return true;
  if (rt() - t0 > DEADLINE) {
    __hip_atomic_store((GAS unsigned*)(ws + OFF_TMO), call, __ATOMIC_RELAXED, __HIP_MEMORY_SCOPE_AGENT);
    return true;
  }
  return false;
}

__global__ void __launch_bounds__(NTHR, 2) fused(Params P) {
  extern __shared__ __attribute__((aligned(16))) char smem[];
  const unsigned wg = blockIdx.x;
  const unsigned lane = threadIdx.x & 63;
  const unsigned wave = rfl(threadIdx.x >> 6);
  const unsigned rank = P.rank;
  char* const ws = P.ws;
  const bool is_gemm = wg < NGEMM;
  const unsigned lds0 = (unsigned)(size_t)(LAS char*)smem;

  if (!is_gemm) {
    const unsigned gw = wave * 128 + (wg - NGEMM);
    if (gw >= 7 * (W1 > W2 ? W1 : W2)) return;
  } else {
    if (threadIdx.x < 128) lds_w32(lds0 + L_CTRL + threadIdx.x * 4, 0u);
    WAITL();
    __builtin_amdgcn_s_barrier();
  }
  STAMP(0);

  GAS unsigned* ctrp = (GAS unsigned*)(ws + OFF_CTR + (wg * 8 + wave) * 4);
  const unsigned ctr = __hip_atomic_load(ctrp, __ATOMIC_RELAXED, __HIP_MEMORY_SCOPE_AGENT);
  char* peer_v = P.ws_all[0];
#pragma unroll
  for (int q = 1; q < NR; ++q) peer_v = (lane & 7) == (unsigned)q ? P.ws_all[q] : peer_v;

  if (is_gemm) {
    const unsigned l31 = lane & 31, kh = lane >> 5;
    const bool isw = wave & 1;
    const unsigned tile = wg >> 2, sub = wg & 3;
    const bool mine = isw ? ((tile >> 3) == (rank & 3)) : ((tile >> 4) == (rank >> 2));
    const unsigned lt = isw ? (tile & 7) : (tile & 15);
    const bool fl_on = lane < 8 && lane != rank && (isw ? ((lane & 3) == (tile >> 3)) : ((lane >> 2) == (tile >> 4)));
    unsigned call = 0;
    if (wave >= 2) __builtin_amdgcn_s_sleep(16);
#pragma unroll
    for (int ps = 0; ps < 2; ++ps) {
      const unsigned c = ps * 4 + (wave >> 1);
      f32x4 v[8];
      if (!isw) {
        const GAS float* src = (const GAS float*)P.x + (size_t)(tile * 128 + sub * 32 + l31) * KS + c * 64 + 4 * kh;
#pragma unroll
        for (int t = 0; t < 8; ++t) v[t] = *(const GAS f32x4*)(src + 8 * t);
      } else {
        const unsigned a = 16 * ((l31 >> 2) & 1) + 4 * (l31 >> 3) + (l31 & 3);
        const GAS float* src = (const GAS float*)P.w + (size_t)(c * 64 + 16 * sub + 4 * kh) * NN + tile * 128 + 4 * a;
#pragma unroll
        for (int q = 0; q < 8; ++q) v[q] = *(const GAS f32x4*)(src + (size_t)(8 * (q >> 2) + (q & 3)) * NN);
      }
      if (ps == 0) {
        call = rfl(ctr) + 1;
        if (lane == 0) __hip_atomic_store(ctrp, call, __ATOMIC_RELAXED, __HIP_MEMORY_SCOPE_AGENT);
      }
      u32x4 f[4];
      if (!isw) {
#pragma unroll
        for (int kb = 0; kb < 4; ++kb) {
          const f32x4 a = v[2 * kb], b = v[2 * kb + 1];
          f[kb] = u32x4{pk_bf16(a[0], a[1]), pk_bf16(a[2], a[3]), pk_bf16(b[0], b[1]), pk_bf16(b[2], b[3])};
        }
      } else {
#pragma unroll
        for (int rb = 0; rb < 4; ++rb)
          f[rb] = u32x4{pk_bf16(v[0][rb], v[1][rb]), pk_bf16(v[2][rb], v[3][rb]),
                        pk_bf16(v[4][rb], v[5][rb]), pk_bf16(v[6][rb], v[7][rb])};
      }
      char* e = ws + (isw ? OFF_EXPW : OFF_EXPX) + (c * 128 + wg) * 4096;
      st16<0>(e, lane * 16, f[0]); st16<1024>(e, lane * 16, f[1]); st16<2048>(e, lane * 16, f[2]); st16<3072>(e, lane * 16, f[3]);
      const unsigned step = rank * 8 + c;
      if (mine) {
        char* d = ws + (isw ? OFF_INW : OFF_INX) + (lt * 64 + step) * 16384 + sub * 4096;
        st16<0>(d, lane * 16, f[0]); st16<1024>(d, lane * 16, f[1]); st16<2048>(d, lane * 16, f[2]); st16<3072>(d, lane * 16, f[3]);
      }
      WAITV(0);
      if (fl_on) st8v(peer_v + OFF_EXPF + ((rank * 8 + c) * 96 + (isw ? 0 : 32) + lt * 4 + sub) * 8, u32x2{0u, call});
      if (mine && lane == 0) st4v(ws + (isw ? OFF_LOCW : OFF_LOCX) + ((lt * 64 + step) * 4 + sub) * 4, call);
      STAMP(1 + ps);
    }

    const unsigned xcd = wg & 7, idx = wg >> 3;
    const unsigned tt = 4 * (xcd >> 1) + (idx >> 2), uu = 4 * (xcd & 1) + (idx & 3);

    if (wave < 4) {
      f32x16 acc[4];
#pragma unroll
      for (int rb = 0; rb < 4; ++rb)
#pragma unroll
        for (int r = 0; r < 16; ++r) acc[rb][r] = 0.f;
#pragma unroll 1
      for (unsigned i = 0; i < 64; ++i) {
        for (;;) {
          const u32x2 fu = lds_r64(lds0 + L_FULL);
          if (rfl(fu[0]) > i && rfl(fu[1]) > i) break;
          __builtin_amdgcn_s_sleep(1);
        }
        const LAS char* slot = (const LAS char*)smem + (i % NS) * 32768 + lane * 16;
        bf16x8 xf[4], wf[4][4];
#pragma unroll
        for (int kb = 0; kb < 4; ++kb) {
          xf[kb] = *(const LAS bf16x8*)(slot + (wave * 4 + kb) * 1024);
#pragma unroll
          for (int rb = 0; rb < 4; ++rb) wf[kb][rb] = *(const LAS bf16x8*)(slot + 16384 + (kb * 4 + rb) * 1024);
        }
        __builtin_amdgcn_sched_barrier(0);
#pragma unroll
        for (int kb = 0; kb < 3; ++kb)
#pragma unroll
          for (int rb = 0; rb < 4; ++rb) acc[rb] = __builtin_amdgcn_mfma_f32_32x32x16_bf16(wf[kb][rb], xf[kb], acc[rb], 0, 0, 0);
        __builtin_amdgcn_sched_barrier(0);
        WAITL();
        lds_w32(lds0 + L_CONS + wave * 4, i + 1);
        __builtin_amdgcn_sched_barrier(0);
#pragma unroll
        for (int rb = 0; rb < 4; ++rb) acc[rb] = __builtin_amdgcn_mfma_f32_32x32x16_bf16(wf[3][rb], xf[3], acc[rb], 0, 0, 0);
      }
      STAMP(5);
      float mf[4];
#pragma unroll
      for (int rb = 0; rb < 4; ++rb) {
        mf[rb] = 0.f;
#pragma unroll
        for (int r = 0; r < 16; r += 2)
          mf[rb] = __builtin_fmaxf(__builtin_fmaxf(__builtin_fabsf(acc[rb][r]), __builtin_fabsf(acc[rb][r + 1])), mf[rb]);
      }
      unsigned m = umax(umax(f2u(mf[0]), f2u(mf[1])), umax(f2u(mf[2]), f2u(mf[3])));
      m = wave_max(m);
      {
        unsigned arr = 0;
        if (lane == 0)
          asm volatile("ds_max_u32 %1, %2\n\tds_add_rtn_u32 %0, %1, %3 offset:4\n\ts_waitcnt lgkmcnt(0)"
                       : "=v"(arr) : "v"(lds0 + L_WMAX), "v"(m), "v"(1u) : "memory");
        if (rfl(arr) == 3) {
          const unsigned wm = lds_r32(lds0 + L_WMAX);
          const unsigned fail = rfl(lds_r32(lds0 + L_FAIL));
          if (lane < 8 && !fail) st8v(peer_v + OFF_AMAX + (rank * 128 + wg) * 8, u32x2{wm, call});
        }
      }
      STAMP(6);
      u32x2 g;
      for (;;) {
        g = lds_r64(lds0 + L_GMAX);
        if (rfl(g[1]) != 0) break;
        __builtin_amdgcn_s_sleep(1);
      }
      STAMP(7);
      const bool ok = rfl(g[1]) == 1;
      const float amax = u2f(g[0]);
      const float inv = amax > 0.f ? 127.0f / amax : 0.f;
      u32x4 q[4];
#pragma unroll
      for (int r = 0; r < 16; ++r) {
        unsigned b[4];
#pragma unroll
        for (int rb = 0; rb < 4; ++rb) b[rb] = f2u(__builtin_fmaf(acc[rb][r], inv, 12582912.0f));
        const unsigned lo = __builtin_amdgcn_perm(b[1], b[0], 0x0c0c0400u);
        const unsigned hi = __builtin_amdgcn_perm(b[3], b[2], 0x04000c0cu);
        q[r >> 2][r & 3] = ok ? (lo | hi) : 0x80808080u;
      }
      LAS char* tb = (LAS char*)smem + wave * 4096;
#pragma unroll
      for (int jj = 0; jj < 4; ++jj) *(LAS u32x4*)(tb + l31 * 128 + (((4 * kh + jj) ^ (l31 & 7)) * 16)) = q[jj];
      WAITL();
#pragma unroll
      for (int jj = 0; jj < 4; ++jj) {
        const unsigned row = 8 * jj + (lane >> 3);
        q[jj] = *(const LAS u32x4*)(tb + row * 128 + (((lane & 7) ^ (row & 7)) * 16));
      }
      const unsigned unit = wg * 4 + wave;
      if (ok) {
        char* e = ws + OFF_EXPO + unit * 4096;
        st16<0>(e, lane * 16, q[0]); st16<1024>(e, lane * 16, q[1]); st16<2048>(e, lane * 16, q[2]); st16<3072>(e, lane * 16, q[3]);
        WAITV(0);
        if (lane < 8 && lane != rank) st8v(peer_v + OFF_OUTF + (rank * 512 + unit) * 8, u32x2{0u, call});
      }
      STAMP(8);
      char* o = P.out + (size_t)((rank >> 2) * 2048 + tt * 128 + wave * 32) * NN + (rank & 3) * 1024 + uu * 128;
      const unsigned vo = (lane >> 3) * NN + (lane & 7) * 16;
      st16<0>(o, vo, q[0]); st16<0>(o, vo + 8 * NN, q[1]); st16<0>(o, vo + 16 * NN, q[2]); st16<0>(o, vo + 24 * NN, q[3]);
      STAMP(9);
    } else if (wave == 4) {
      char* fx = ws + OFF_LOCX + (tt * 64 + lane) * 16;
      char* fw = ws + OFF_LOCW + (uu * 64 + lane) * 16;
      char* ft = ws + OFF_TMO;
      u64 done = 0;
      unsigned n = 0;
      const u64 t0 = rt();
      while (n < 64) {
        u32x4 a, b;
        unsigned tmo;
        asm volatile("global_load_dwordx4 %0, %3, off sc0 sc1\n\tglobal_load_dwordx4 %1, %4, off sc0 sc1\n\t"
                     "global_load_dword %2, %5, off sc1\n\ts_waitcnt vmcnt(0)"
                     : "=&v"(a), "=&v"(b), "=&v"(tmo) : "v"(fx), "v"(fw), "v"(ft) : "memory");
        const bool rdy = a[0] >= call && a[1] >= call && a[2] >= call && a[3] >= call &&
                         b[0] >= call && b[1] >= call && b[2] >= call && b[3] >= call;
        u64 mask = __ballot(rdy) & ~done;
        if (mask == 0) {
          if (give_up(ws, rfl(tmo), call, t0)) {
            lds_w32(lds0 + L_FAIL, 1u);
            mask = ~done;
          } else {
            __builtin_amdgcn_s_sleep(2);
            continue;
          }
        }
        const unsigned pos = n + __builtin_amdgcn_mbcnt_hi((unsigned)(mask >> 32), __builtin_amdgcn_mbcnt_lo((unsigned)mask, 0u));
        if ((mask >> lane) & 1) lds_w32(lds0 + L_ORDER + pos * 4, lane);
        n += __builtin_popcountll(mask);
        done |= mask;
        WAITL();
        lds_w32(lds0 + L_CNT, n);
      }
      STAMP(5);
    } else if (wave < 7) {
      const unsigned sel = wave - 5;
      char* base = ws + (sel ? OFF_INW + uu * 64 * 16384 : OFF_INX + tt * 64 * 16384);
      const unsigned vo0 = lane * 16, vo1 = vo0 + 4096, vo2 = vo0 + 8192, vo3 = vo0 + 12288;
      unsigned issued = 0, landed = 0;
      while (landed < 64) {
        unsigned cnt;
        u32x4 cs;
        asm volatile("ds_read_b32 %0, %2\n\tds_read_b128 %1, %2 offset:32\n\ts_waitcnt lgkmcnt(0)"
                     : "=&v"(cnt), "=&v"(cs) : "v"(lds0 + L_CNT) : "memory");
        const unsigned c01 = cs[0] < cs[1] ? cs[0] : cs[1], c23 = cs[2] < cs[3] ? cs[2] : cs[3];
        const unsigned cmin = rfl(c01 < c23 ? c01 : c23) + NS, cn = rfl(cnt);
        const unsigned avail = cn < cmin ? cn : cmin;
        if (issued < avail && issued - landed < 3) {
          const unsigned step = rfl(lds_r32(lds0 + L_ORDER + issued * 4));
          char* src = base + step * 16384;
          const unsigned dst = lds0 + (issued % NS) * 32768 + sel * 16384;
          unsigned keep;
          asm volatile(
              "s_nop 4\n\ts_mov_b32 %0, m0\n\t"
              "s_mov_b32 m0, %6\n\ts_nop 0\n\t"
              "global_load_lds_dwordx4 %1, %5 sc0 sc1\n\tglobal_load_lds_dwordx4 %1, %5 offset:1024 sc0 sc1\n\t"
              "global_load_lds_dwordx4 %1, %5 offset:2048 sc0 sc1\n\tglobal_load_lds_dwordx4 %1, %5 offset:3072 sc0 sc1\n\t"
              "s_add_u32 m0, m0, 0x1000\n\ts_nop 0\n\t"
              "global_load_lds_dwordx4 %2, %5 sc0 sc1\n\tglobal_load_lds_dwordx4 %2, %5 offset:1024 sc0 sc1\n\t"
              "global_load_lds_dwordx4 %2, %5 offset:2048 sc0 sc1\n\tglobal_load_lds_dwordx4 %2, %5 offset:3072 sc0 sc1\n\t"
              "s_add_u32 m0, m0, 0x1000\n\ts_nop 0\n\t"
              "global_load_lds_dwordx4 %3, %5 sc0 sc1\n\tglobal_load_lds_dwordx4 %3, %5 offset:1024 sc0 sc1\n\t"
              "global_load_lds_dwordx4 %3, %5 offset:2048 sc0 sc1\n\tglobal_load_lds_dwordx4 %3, %5 offset:3072 sc0 sc1\n\t"
              "s_add_u32 m0, m0, 0x1000\n\ts_nop 0\n\t"
              "global_load_lds_dwordx4 %4, %5 sc0 sc1\n\tglobal_load_lds_dwordx4 %4, %5 offset:1024 sc0 sc1\n\t"
              "global_load_lds_dwordx4 %4, %5 offset:2048 sc0 sc1\n\tglobal_load_lds_dwordx4 %4, %5 offset:3072 sc0 sc1\n\t"
              "s_mov_b32 m0, %0"
              : "=&s"(keep) : "v"(vo0), "v"(vo1), "v"(vo2), "v"(vo3), "s"(src), "s"(dst) : "memory", "scc");
          ++issued;
        } else if (issued > landed) {
          const unsigned fl = issued - landed;
          if (fl == 1) WAITV(0); else if (fl == 2) WAITV(16); else WAITV(32);
          ++landed;
          lds_w32(lds0 + L_FULL + sel * 4, landed);
        } else {
          __builtin_amdgcn_s_sleep(1);
        }
      }
      STAMP(5);
    } else {
      for (;;) {
        if (rfl(lds_r32(lds0 + L_CNT)) >= 64) break;
        __builtin_amdgcn_s_sleep(4);
      }
      char* ga = ws + OFF_AMAX;
      char* ft = ws + OFF_TMO;
      const unsigned vo0 = lane * 16, vo1 = vo0 + 4096;
      const u64 t0 = rt();
      unsigned state, m;
      for (;;) {
        u32x4 a[8];
        unsigned tmo;
        asm volatile("s_nop 4\n\t"
                     "global_load_dwordx4 %0, %9, %11 sc0 sc1\n\tglobal_load_dwordx4 %1, %9, %11 offset:1024 sc0 sc1\n\t"
                     "global_load_dwordx4 %2, %9, %11 offset:2048 sc0 sc1\n\tglobal_load_dwordx4 %3, %9, %11 offset:3072 sc0 sc1\n\t"
                     "global_load_dwordx4 %4, %10, %11 sc0 sc1\n\tglobal_load_dwordx4 %5, %10, %11 offset:1024 sc0 sc1\n\t"
                     "global_load_dwordx4 %6, %10, %11 offset:2048 sc0 sc1\n\tglobal_load_dwordx4 %7, %10, %11 offset:3072 sc0 sc1\n\t"
                     "global_load_dword %8, %12, off sc1\n\ts_waitcnt vmcnt(0)"
                     : "=&v"(a[0]), "=&v"(a[1]), "=&v"(a[2]), "=&v"(a[3]), "=&v"(a[4]), "=&v"(a[5]), "=&v"(a[6]), "=&v"(a[7]), "=&v"(tmo)
                     : "v"(vo0), "v"(vo1), "s"(ga), "v"(ft) : "memory");
        bool rdy = true;
        m = 0;
#pragma unroll
        for (int k = 0; k < 8; ++k) {
          rdy = rdy && a[k][1] >= call && a[k][3] >= call;
          const unsigned mm = a[k][0] > a[k][2] ? a[k][0] : a[k][2];
          m = m > mm ? m : mm;
        }
        if (__ballot(rdy) == ~0ull) { state = 1; break; }
        if (give_up(ws, rfl(tmo), call, t0)) { state = 2; break; }
        __builtin_amdgcn_s_sleep(1);
      }
      m = wave_max(m);
      lds_w64(lds0 + L_GMAX, u32x2{m, state});
      STAMP(5);
    }
  } else {
    const unsigned call = rfl(ctr) + 1;
    if (lane == 0) __hip_atomic_store(ctrp, call, __ATOMIC_RELAXED, __HIP_MEMORY_SCOPE_AGENT);
    const unsigned gw = wave * 128 + (wg - NGEMM);
    const unsigned link = gw % 7, slot = gw / 7;
    const unsigned p = (rank + 1 + link) & 7;
    char* const pws = mkptr(rdl((unsigned)(u64)peer_v, p), rdl((unsigned)((u64)peer_v >> 32), p));
    char* const ft = ws + OFF_TMO;
    const unsigned vo = lane * 16;
    bool failed = false;
    if (slot < W1) {
      const unsigned n = slot + W1 * lane;
      const unsigned nunits = (768 - slot + W1 - 1) / W1;
      const unsigned nn = n < 768 ? n : 0;
      const unsigned c = nn / 96, r = nn % 96, sub = r & 3;
      const bool isw = r < 32;
      const unsigned lt = isw ? (r >> 2) : ((((r - 32) >> 2) + (link * 16) / 7) & 15);
      const unsigned gt = isw ? (rank & 3) * 8 + lt : (rank >> 2) * 16 + lt;
      const unsigned step = p * 8 + c;
      const unsigned d_src = (isw ? OFF_EXPW : OFF_EXPX) + (c * 128 + gt * 4 + sub) * 4096;
      const unsigned d_dst = (isw ? OFF_INW : OFF_INX) + (lt * 64 + step) * 16384 + sub * 4096;
      const unsigned d_flag = (isw ? OFF_LOCW : OFF_LOCX) + ((lt * 64 + step) * 4 + sub) * 4;
      char* const ef = ws + OFF_EXPF + (step * 96 + (isw ? 0 : 32) + lt * 4 + sub) * 8;
      u64 ready = 0;
      const u64 t0 = rt();
      bool pend = false;
      unsigned pflag = 0;
      for (unsigned i = 0; i < nunits; ++i) {
        while (!((ready >> i) & 1)) {
          u32x2 g;
          unsigned tmo;
          asm volatile("global_load_dwordx2 %0, %2, off sc0 sc1\n\tglobal_load_dword %1, %3, off sc1\n\ts_waitcnt vmcnt(0)"
                       : "=&v"(g), "=&v"(tmo) : "v"(ef), "v"(ft) : "memory");
          if (pend) { if (lane == 0) st4v(ws + pflag, call); pend = false; }
          ready = __ballot(g[1] >= call);
          if ((ready >> i) & 1) break;
          if (give_up(ws, rfl(tmo), call, t0)) { failed = true; break; }
          __builtin_amdgcn_s_sleep(1);
        }
        if (failed) break;
        char* s = pws + rdl(d_src, i);
        u32x4 a, b, cc, d;
        asm volatile("s_nop 4\n\tglobal_load_dwordx4 %0, %4, %5 sc0 sc1\n\tglobal_load_dwordx4 %1, %4, %5 offset:1024 sc0 sc1\n\t"
                     "global_load_dwordx4 %2, %4, %5 offset:2048 sc0 sc1\n\tglobal_load_dwordx4 %3, %4, %5 offset:3072 sc0 sc1\n\t"
                     "s_waitcnt vmcnt(4)"
                     : "=&v"(a), "=&v"(b), "=&v"(cc), "=&v"(d) : "v"(vo), "s"(s) : "memory");
        if (pend) { if (lane == 0) st4v(ws + pflag, call); }
        asm volatile("s_waitcnt vmcnt(0)" : "+v"(a), "+v"(b), "+v"(cc), "+v"(d) :: "memory");
        char* o = ws + rdl(d_dst, i);
        st16<0>(o, vo, a); st16<1024>(o, vo, b); st16<2048>(o, vo, cc); st16<3072>(o, vo, d);
        pend = true;
        pflag = rdl(d_flag, i);
      }
      STAMP(3);
      WAITV(0);
      if (pend && lane == 0) st4v(ws + pflag, call);
      STAMP(4);
    }
    if (slot < W2) {
      const unsigned n = slot + W2 * lane;
      const unsigned nunits = (512 - slot + W2 - 1) / W2;
      const unsigned nn = n < 512 ? n : 0;
      const unsigned sw = nn >> 2, sx = sw & 7, si = sw >> 3;
      const unsigned st = 4 * (sx >> 1) + (si >> 2), su = 4 * (sx & 1) + (si & 3);
      const unsigned d_src = OFF_EXPO + nn * 4096;
      const unsigned d_dst = ((p >> 2) * 2048 + st * 128 + (nn & 3) * 32) * NN + (p & 3) * 1024 + su * 128;
      char* const ef = ws + OFF_OUTF + (p * 512 + nn) * 8;
      const unsigned wo = (lane >> 3) * NN + (lane & 7) * 16;
      u64 todo = nunits >= 64 ? ~0ull : ((1ull << nunits) - 1);
      u64 ready = 0;
      const u64 t0 = rt();
      bool first = true;
      while (todo) {
        while (!(ready & todo) && !failed) {
          u32x2 g;
          unsigned tmo;
          asm volatile("global_load_dwordx2 %0, %2, off sc0 sc1\n\tglobal_load_dword %1, %3, off sc1\n\ts_waitcnt vmcnt(0)"
                       : "=&v"(g), "=&v"(tmo) : "v"(ef), "v"(ft) : "memory");
          ready = __ballot(g[1] >= call);
          if (ready & todo) break;
          if (give_up(ws, rfl(tmo), call, t0)) { failed = true; break; }
          __builtin_amdgcn_s_sleep(1);
        }
        first = false;
        const unsigned i = __builtin_ctzll(failed ? todo : (ready & todo));
        todo &= ~(1ull << i);
        u32x4 a, b, cc, d;
        if (!failed) {
          char* s = pws + rdl(d_src, i);
          asm volatile("s_nop 4\n\tglobal_load_dwordx4 %0, %4, %5 sc0 sc1\n\tglobal_load_dwordx4 %1, %4, %5 offset:1024 sc0 sc1\n\t"
                       "global_load_dwordx4 %2, %4, %5 offset:2048 sc0 sc1\n\tglobal_load_dwordx4 %3, %4, %5 offset:3072 sc0 sc1\n\t"
                       "s_waitcnt vmcnt(0)"
                       : "=&v"(a), "=&v"(b), "=&v"(cc), "=&v"(d) : "v"(vo), "s"(s) : "memory");
        } else {
          a = b = cc = d = u32x4{0x80808080u, 0x80808080u, 0x80808080u, 0x80808080u};
        }
        char* o = P.out + rdl(d_dst, i);
        st16<0>(o, wo, a); st16<0>(o, wo + 8 * NN, b); st16<0>(o, wo + 16 * NN, cc); st16<0>(o, wo + 24 * NN, d);
      }
      STAMP(6);
      WAITV(0);
      STAMP(7);
    }
  }
}

}

extern "C" void kernel_launch(int rank, int nranks, ncclComm_t comm, hipStream_t stream,
                              void* const* d_in, const int* in_sizes, int n_in,
                              void* d_out, int out_size, const int* in_dtypes, int out_dtype,
                              void* const* d_out_all, void* const* d_ws_all, size_t ws_bytes) {
  if (nranks != NR || (unsigned)rank >= (unsigned)NR || ws_bytes < WS_NEED || n_in < 2 || out_size != MM * NN ||
      in_sizes[0] != MM * KS || in_sizes[1] != KS * NN) {
    fprintf(stderr, "kernel_launch: unexpected problem shape\n");
    return;
  }
  hipError_t e = hipFuncSetAttribute((const void*)fused, hipFuncAttributeMaxDynamicSharedMemorySize, (int)LDS_BYTES);
  if (e != hipSuccess) { fprintf(stderr, "hipFuncSetAttribute: %s\n", hipGetErrorString(e)); return; }
  Params P;
  P.x = (const float*)d_in[0];
  P.w = (const float*)d_in[1];
  P.out = (char*)d_out;
  P.ws = (char*)d_ws_all[rank];
  for (int q = 0; q < NR; ++q) P.ws_all[q] = (char*)d_ws_all[q];
  P.rank = rank;
  P.pad = 0;
  hipLaunchKernelGGL(fused, dim3(NWG), dim3(NTHR), LDS_BYTES, stream, P);
  e = hipGetLastError();
  if (e != hipSuccess) fprintf(stderr, "launch: %s\n", hipGetErrorString(e));
}
```

```cpp
#include <hip/hip_runtime.h>
#include <rccl/rccl.h>
#include <cstdio>

#define GAS __attribute__((address_space(1)))
#define LAS __attribute__((address_space(3)))
typedef unsigned u32x2 __attribute__((ext_vector_type(2)));
typedef unsigned u32x4 __attribute__((ext_vector_type(4)));
typedef float f32x2 __attribute__((ext_vector_type(2)));
typedef float f32x4 __attribute__((ext_vector_type(4)));
typedef float f32x16 __attribute__((ext_vector_type(16)));
typedef __bf16 bf16x2 __attribute__((ext_vector_type(2)));
typedef __bf16 bf16x8 __attribute__((ext_vector_type(8)));
typedef unsigned short u16x2 __attribute__((ext_vector_type(2)));
typedef unsigned long long u64;

namespace {

constexpr int NR = 8;
constexpr int MM = 4096, NN = 4096, KS = 512;
constexpr int NWG = 256, NTHR = 512, NGEMM = 128;
#define W1 11
#define W2 18
#define NAGG 2
#define SLP2 16
#define NCAND 32
#define PREF 16
#define PSLP 8
#define SLP3 80
#define NPW 4
#define REL "3"
#define FSLP "4"
#define SLP4 32
#define OLDM 0x05u
#define LASTQ 1
#define BACKL 6
#define ZA 40
#define ZB 120
#define XSLP 16
#define ASL0 12
#define FASL 64
#define NSKIP 20
#define ZG 48
#define LV4 1
#define NREP 8
#define RELA "5"
#define JS2 32
#define ZS 21
#define RELQ "1"
#define WM 13
#define XPOS2 575
#define NPUSH 8
#define ZR 124
#define SGG 4
#define XPOS 490
static_assert(NPW <= 4 && W1 >= 10 && W1 <= 12 && W1 <= W2 && W2 <= 18 && NCAND <= 40 && ZA >= NCAND && ZA % SGG == 0 && ZB % SGG == 0 && SGG >= 2 && ZS <= NCAND, "puller counts");
constexpr unsigned NF = NPUSH + W1 < 16 ? NPUSH + W1 : 16;
constexpr unsigned NSG = (ZB - ZA) / 2;
constexpr unsigned N1 = 384, NU = N1 + 512 + NSG;
constexpr u64 DEADLINE = 150000000ull;

constexpr unsigned OFF_TMO = 0;
constexpr unsigned OFF_ALLC = 256;
constexpr unsigned OFF_MIDC = 512;
constexpr unsigned OFF_TK = 1024;
constexpr unsigned OFF_BFF = 2048;
constexpr unsigned OFF_CTR = 4096;
constexpr unsigned OFF_OUTC = 12288;
constexpr unsigned OFF_GO = 13312;
constexpr unsigned OFF_SGF = 16384;
constexpr unsigned OFF_SGC = 24576;
constexpr unsigned OFF_LOCX = 32768;
constexpr unsigned OFF_LOCW = OFF_LOCX + 8192;
constexpr unsigned OFF_EXPF = 65536;
constexpr unsigned OFF_OUTF = 69632;
constexpr unsigned OFF_AMAX1 = 81920;
constexpr unsigned OFF_AMAX2 = 98304;
constexpr unsigned OFF_EXPE = 106496;
constexpr unsigned OFF_EXPG = 172032;
constexpr unsigned IU = 6144;
constexpr unsigned OU = 4224;
constexpr unsigned OFF_EXPI = 1u << 20;
constexpr unsigned OFF_EXPP = 8u << 20;
constexpr unsigned OFF_EXPO = 13u << 20;
constexpr unsigned OFF_INX = 16u << 20;
constexpr unsigned OFF_INW = 32u << 20;
constexpr unsigned OFF_EXPQ = 40u << 20;
constexpr unsigned OFF_SGN = 42u << 20;
constexpr size_t WS_NEED = 44u << 20;

constexpr unsigned L_CTRL = 8 * 16384;
constexpr unsigned L_CF = L_CTRL;
constexpr unsigned L_LORD = L_CTRL + 512;
constexpr unsigned L_REL = L_CTRL + 768;
constexpr unsigned L_WMAX = L_CTRL + 1024;
constexpr unsigned L_GMAX = L_CTRL + 1040;
constexpr unsigned L_FAIL = L_CTRL + 1056;
constexpr unsigned L_OARR = L_CTRL + 1060;
constexpr unsigned L_BARR = L_CTRL + 1064;
constexpr unsigned L_MARR = L_CTRL + 1068;
constexpr unsigned L_ENC = L_CTRL + 2048;
constexpr unsigned LDS_BYTES = L_ENC + 4 * 7168;
constexpr unsigned NFS = 14;
constexpr unsigned L_FCNT = L_CTRL;
constexpr unsigned L_FDONE = L_CTRL + 8;
constexpr unsigned L_TBL = L_CTRL + 12;
constexpr unsigned L_LOCK = L_CTRL + 16;
constexpr unsigned L_FDESC = L_CTRL + 32;
constexpr unsigned L_DESC = 0;

struct Params {
  const float* x;
  const float* w;
  char* out;
  char* ws;
  char* ws_all[NR];
  int rank;
  int pad;
};

__device__ __forceinline__ u64 rt() { return __builtin_amdgcn_s_memrealtime(); }
__device__ __forceinline__ unsigned rfl(unsigned v) { return __builtin_amdgcn_readfirstlane(v); }
__device__ __forceinline__ unsigned sreg(unsigned v) {
  unsigned r;
  asm("s_nop 0\n\tv_readfirstlane_b32 %0, %1\n\ts_nop 4" : "=s"(r) : "v"(v));
  return r;
}
__device__ __forceinline__ unsigned rdl(unsigned v, unsigned l) { return __builtin_amdgcn_readlane(v, l); }
__device__ __forceinline__ char* lane_ptr(char* v, unsigned l) {
  return (char*)(((u64)rdl((unsigned)((u64)v >> 32), l) << 32) | rdl((unsigned)(u64)v, l));
}
__device__ __forceinline__ unsigned f2u(float f) { return __builtin_bit_cast(unsigned, f); }
__device__ __forceinline__ float u2f(unsigned u) { return __builtin_bit_cast(float, u); }
__device__ __forceinline__ unsigned umax(unsigned a, unsigned b) { return a > b ? a : b; }
__device__ __forceinline__ unsigned bfi(unsigned m, unsigned a, unsigned b) { return (a & m) | (b & ~m); }
__device__ __forceinline__ unsigned rotr(unsigned a, unsigned s) { return __builtin_amdgcn_alignbit(a, a, s); }
__device__ __forceinline__ unsigned perm(unsigned hi, unsigned lo, unsigned sel) { return __builtin_amdgcn_perm(hi, lo, sel); }
__device__ __forceinline__ unsigned pk_add(unsigned a, unsigned b) {
  return __builtin_bit_cast(unsigned, (u16x2)(__builtin_bit_cast(u16x2, a) + __builtin_bit_cast(u16x2, b)));
}
__device__ __forceinline__ unsigned pk_sub(unsigned a, unsigned b) {
  return __builtin_bit_cast(unsigned, (u16x2)(__builtin_bit_cast(u16x2, a) - __builtin_bit_cast(u16x2, b)));
}
__device__ __forceinline__ unsigned pk_bf16(float a, float b) {
  f32x2 f = {a, b};
  return __builtin_bit_cast(unsigned, __builtin_convertvector(f, bf16x2));
}

#define WAITV(n) asm volatile("s_waitcnt vmcnt(" #n ")" ::: "memory")
#define WAITL() asm volatile("s_waitcnt lgkmcnt(0)" ::: "memory")
#define SC " sc0 sc1"

__device__ __forceinline__ void st16x4(char* sb, unsigned vo, u32x4 a, u32x4 b, u32x4 c, u32x4 d) {
  asm volatile("s_nop 4\n\tglobal_store_dwordx4 %0, %1, %5" SC "\n\tglobal_store_dwordx4 %0, %2, %5 offset:1024" SC
               "\n\tglobal_store_dwordx4 %0, %3, %5 offset:2048" SC "\n\tglobal_store_dwordx4 %0, %4, %5 offset:3072" SC "\n\ts_nop 1"
               :: "v"(vo), "v"(a), "v"(b), "v"(c), "v"(d), "s"(sb) : "memory");
}
__device__ __forceinline__ void st16r4(char* sb, unsigned v0, unsigned v1, unsigned v2, unsigned v3, u32x4 a, u32x4 b, u32x4 c, u32x4 d) {
  asm volatile("s_nop 4\n\tglobal_store_dwordx4 %0, %4, %8" SC "\n\tglobal_store_dwordx4 %1, %5, %8" SC
               "\n\tglobal_store_dwordx4 %2, %6, %8" SC "\n\tglobal_store_dwordx4 %3, %7, %8" SC "\n\ts_nop 1"
               :: "v"(v0), "v"(v1), "v"(v2), "v"(v3), "v"(a), "v"(b), "v"(c), "v"(d), "s"(sb) : "memory");
}
__device__ __forceinline__ void st16(char* sb, unsigned vo, u32x4 a) {
  asm volatile("s_nop 4\n\tglobal_store_dwordx4 %0, %1, %2" SC "\n\ts_nop 1" :: "v"(vo), "v"(a), "s"(sb) : "memory");
}
__device__ __forceinline__ void st8(char* sb, unsigned vo, u32x2 a) {
  asm volatile("s_nop 4\n\tglobal_store_dwordx2 %0, %1, %2" SC "\n\ts_nop 1" :: "v"(vo), "v"(a), "s"(sb) : "memory");
}
__device__ __forceinline__ void st4(char* sb, unsigned vo, unsigned a) {
  asm volatile("s_nop 4\n\tglobal_store_dword %0, %1, %2" SC "\n\ts_nop 1" :: "v"(vo), "v"(a), "s"(sb) : "memory");
}
__device__ __forceinline__ void st8v(char* p, u32x2 v) {
  asm volatile("global_store_dwordx2 %0, %1, off" SC "\n\ts_nop 1" :: "v"(p), "v"(v) : "memory");
}
__device__ __forceinline__ void st4v(char* p, unsigned v) {
  asm volatile("global_store_dword %0, %1, off" SC "\n\ts_nop 1" :: "v"(p), "v"(v) : "memory");
}
__device__ __forceinline__ unsigned lds_r32(unsigned a) {
  unsigned v;
  asm volatile("ds_read_b32 %0, %1\n\ts_waitcnt lgkmcnt(0)" : "=v"(v) : "v"(a) : "memory");
  return v;
}
__device__ __forceinline__ u32x2 lds_r64(unsigned a) {
  u32x2 v;
  asm volatile("ds_read_b64 %0, %1\n\ts_waitcnt lgkmcnt(0)" : "=v"(v) : "v"(a) : "memory");
  return v;
}
__device__ __forceinline__ void lds_w32(unsigned a, unsigned v) {
  asm volatile("ds_write_b32 %0, %1" :: "v"(a), "v"(v) : "memory");
}
__device__ __forceinline__ void lds_w64(unsigned a, u32x2 v) {
  asm volatile("ds_write_b64 %0, %1" :: "v"(a), "v"(v) : "memory");
}
#define CFENCE() asm volatile("" ::: "memory")

#define DPP0(v, ctrl, rm, bc) ((unsigned)__builtin_amdgcn_update_dpp(0, (int)(v), ctrl, rm, 0xf, bc))
__device__ __forceinline__ unsigned wave_max(unsigned v) {
  v = umax(v, DPP0(v, 0x111, 0xf, true));
  v = umax(v, DPP0(v, 0x112, 0xf, true));
  v = umax(v, DPP0(v, 0x114, 0xf, true));
  v = umax(v, DPP0(v, 0x118, 0xf, true));
  v = umax(v, DPP0(v, 0x142, 0xa, false));
  v = umax(v, DPP0(v, 0x143, 0xc, false));
  return rdl(v, 63);
}
__device__ __forceinline__ unsigned wave_scan(unsigned v) {
  v += DPP0(v, 0x111, 0xf, true);
  v += DPP0(v, 0x112, 0xf, true);
  v += DPP0(v, 0x114, 0xf, true);
  v += DPP0(v, 0x118, 0xf, true);
  v += DPP0(v, 0x142, 0xa, false);
  v += DPP0(v, 0x143, 0xc, false);
  return v;
}


__device__ __forceinline__ bool give_up(char* ws, unsigned tmo, unsigned call, u64 t0) {
  if (tmo == call) return true;
  if (rt() - t0 > DEADLINE) {
    __hip_atomic_store((GAS unsigned*)(ws + OFF_TMO), call, __ATOMIC_RELAXED, __HIP_MEMORY_SCOPE_AGENT);
    return true;
  }
  return false;
}
__device__ __forceinline__ void poll128(char* p, unsigned call, u64& r0, u64& r1, unsigned& tmo) {
  u32x2 g;
  asm volatile("global_load_dwordx2 %0, %1, off" SC "\n\ts_waitcnt vmcnt(0)" : "=&v"(g) : "v"(p) : "memory");
  r0 = __ballot(g[0] >= call);
  r1 = __ballot(g[1] >= call);
  tmo = rdl(g[0], 63);
}
__device__ __forceinline__ unsigned rank_in(u64 m) {
  return __builtin_amdgcn_mbcnt_hi((unsigned)(m >> 32), __builtin_amdgcn_mbcnt_lo((unsigned)m, 0u));
}
__device__ __forceinline__ bool bit128(u64 r0, u64 r1, unsigned i) { return (((i & 1) ? r1 : r0) >> (i >> 1)) & 1; }

#define XSC SC
#define CSLP "1"
#define LSLP 1
#define CONSUME_ASM \
      "s_mov_b32 %4, -1\n" \
      "s_mov_b32 %5, 1\n" \
      "s_mov_b32 %9, 0\n" \
      "s_branch LSPIN3_%=\n" \
      "LOOP0_%=:\n" \
      "s_add_u32 %4, %4, 1\n" \
      "s_sub_u32 %5, %5, 1\n" \
      "v_add_u32 %12, %10, %16\n" \
      "s_cmp_eq_u32 %5, 3\n" \
      "s_cbranch_scc1 LW8_0_%=\n" \
      "s_cmp_eq_u32 %5, 2\n" \
      "s_cbranch_scc1 LW4_0_%=\n" \
      "s_waitcnt vmcnt(0)\n" \
      "LW4_0_%=:\n" \
      "s_waitcnt vmcnt(4)\n" \
      "LW8_0_%=:\n" \
      "s_waitcnt vmcnt(8)\n" \
      "s_waitcnt lgkmcnt(7)\n" \
      "v_mfma_f32_32x32x16_bf16 %0, v[128:131], v[192:195], %0\n" \
      "ds_read_b128 v[160:163], %11 offset:8192\n" \
      "ds_read_b128 v[164:167], %11 offset:9216\n" \
      "s_waitcnt lgkmcnt(8)\n" \
      "v_mfma_f32_32x32x16_bf16 %1, v[132:135], v[192:195], %1\n" \
      "ds_read_b128 v[168:171], %11 offset:10240\n" \
      "ds_read_b128 v[172:175], %11 offset:11264\n" \
      "s_waitcnt lgkmcnt(9)\n" \
      "v_mfma_f32_32x32x16_bf16 %2, v[136:139], v[192:195], %2\n" \
      "ds_read_b128 v[176:179], %11 offset:12288\n" \
      "ds_read_b128 v[180:183], %11 offset:13312\n" \
      "s_waitcnt lgkmcnt(10)\n" \
      "v_mfma_f32_32x32x16_bf16 %3, v[140:143], v[192:195], %3\n" \
      "ds_read_b128 v[184:187], %11 offset:14336\n" \
      "ds_read_b128 v[188:191], %11 offset:15360\n" \
      "s_waitcnt lgkmcnt(8)\n" \
      "v_mfma_f32_32x32x16_bf16 %0, v[144:147], v[196:199], %0\n" \
      "ds_read_b64 v[124:125], %15\n" \
      "v_mfma_f32_32x32x16_bf16 %1, v[148:151], v[196:199], %1\n" \
      "v_mfma_f32_32x32x16_bf16 %2, v[152:155], v[196:199], %2\n" \
      "s_waitcnt lgkmcnt(0)\n" \
      "ds_write_b8 %12, %18\n" \
      "v_mfma_f32_32x32x16_bf16 %3, v[156:159], v[196:199], %3\n" \
      "s_add_u32 %7, %4, 1\n" \
      "v_readlane_b32 %6, v124, %7\n" \
      "s_add_u32 %7, %7, 1\n" \
      "s_lshr_b32 %8, %6, 16\n" \
      "s_cmp_eq_u32 %8, %7\n" \
      "s_cbranch_scc0 LSLOW0_%=\n" \
      "s_bfe_u32 %6, %6, 0x60008\n" \
      "v_readlane_b32 %8, v125, %6\n" \
      "s_lshl_b32 %10, %6, 2\n" \
      "s_and_b32 %6, %6, 7\n" \
      "s_lshl_b32 %6, %6, 14\n" \
      "s_cmp_lg_u32 %8, 0\n" \
      "s_cbranch_scc0 LSLOW0_%=\n" \
      "v_add_u32 %11, %6, %14\n" \
      "v_mfma_f32_32x32x16_bf16 %0, v[160:163], v[200:203], %0\n" \
      "ds_read_b128 v[128:131], %11 offset:0\n" \
      "ds_read_b128 v[132:135], %11 offset:1024\n" \
      "v_mfma_f32_32x32x16_bf16 %1, v[164:167], v[200:203], %1\n" \
      "ds_read_b128 v[136:139], %11 offset:2048\n" \
      "ds_read_b128 v[140:143], %11 offset:3072\n" \
      "v_mfma_f32_32x32x16_bf16 %2, v[168:171], v[200:203], %2\n" \
      "ds_read_b128 v[144:147], %11 offset:4096\n" \
      "ds_read_b128 v[148:151], %11 offset:5120\n" \
      "v_mfma_f32_32x32x16_bf16 %3, v[172:175], v[200:203], %3\n" \
      "ds_read_b128 v[152:155], %11 offset:6144\n" \
      "ds_read_b128 v[156:159], %11 offset:7168\n" \
      "v_mfma_f32_32x32x16_bf16 %0, v[176:179], v[204:207], %0\n" \
      "s_cmp_eq_u32 %5, 1\n" \
      "s_cbranch_scc0 LSK1_%=\n" \
      "s_add_u32 %7, %4, 1\n" \
      "v_readlane_b32 %6, v124, %7\n" \
      "s_add_u32 %7, %7, 1\n" \
      "s_lshr_b32 %8, %6, 16\n" \
      "s_cmp_eq_u32 %8, %7\n" \
      "s_cbranch_scc0 LSK1_%=\n" \
      "s_and_b32 %6, %6, 63\n" \
      "s_lshl_b32 %6, %6, 14\n" \
      "v_add_u32 %12, %6, %13\n" \
      "s_add_u32 %5, %5, 1\n" \
      "global_load_dwordx4 v[208:211], %12, %17 offset:0" XSC "\n" \
      "global_load_dwordx4 v[212:215], %12, %17 offset:1024" XSC "\n" \
      "global_load_dwordx4 v[216:219], %12, %17 offset:2048" XSC "\n" \
      "global_load_dwordx4 v[220:223], %12, %17 offset:3072" XSC "\n" \
      "LSK1_%=:\n" \
      "v_mfma_f32_32x32x16_bf16 %1, v[180:183], v[204:207], %1\n" \
      "s_cmp_eq_u32 %5, 2\n" \
      "s_cbranch_scc0 LSK2_%=\n" \
      "s_add_u32 %7, %4, 2\n" \
      "v_readlane_b32 %6, v124, %7\n" \
      "s_add_u32 %7, %7, 1\n" \
      "s_lshr_b32 %8, %6, 16\n" \
      "s_cmp_eq_u32 %8, %7\n" \
      "s_cbranch_scc0 LSK2_%=\n" \
      "s_and_b32 %6, %6, 63\n" \
      "s_lshl_b32 %6, %6, 14\n" \
      "v_add_u32 %12, %6, %13\n" \
      "s_add_u32 %5, %5, 1\n" \
      "global_load_dwordx4 v[224:227], %12, %17 offset:0" XSC "\n" \
      "global_load_dwordx4 v[228:231], %12, %17 offset:1024" XSC "\n" \
      "global_load_dwordx4 v[232:235], %12, %17 offset:2048" XSC "\n" \
      "global_load_dwordx4 v[236:239], %12, %17 offset:3072" XSC "\n" \
      "LSK2_%=:\n" \
      "v_mfma_f32_32x32x16_bf16 %2, v[184:187], v[204:207], %2\n" \
      "s_cmp_eq_u32 %5, 3\n" \
      "s_cbranch_scc0 LSK3_%=\n" \
      "s_add_u32 %7, %4, 3\n" \
      "v_readlane_b32 %6, v124, %7\n" \
      "s_add_u32 %7, %7, 1\n" \
      "s_lshr_b32 %8, %6, 16\n" \
      "s_cmp_eq_u32 %8, %7\n" \
      "s_cbranch_scc0 LSK3_%=\n" \
      "s_and_b32 %6, %6, 63\n" \
      "s_lshl_b32 %6, %6, 14\n" \
      "v_add_u32 %12, %6, %13\n" \
      "s_add_u32 %5, %5, 1\n" \
      "global_load_dwordx4 v[240:243], %12, %17 offset:0" XSC "\n" \
      "global_load_dwordx4 v[244:247], %12, %17 offset:1024" XSC "\n" \
      "global_load_dwordx4 v[248:251], %12, %17 offset:2048" XSC "\n" \
      "global_load_dwordx4 v[252:255], %12, %17 offset:3072" XSC "\n" \
      "LSK3_%=:\n" \
      "v_mfma_f32_32x32x16_bf16 %3, v[188:191], v[204:207], %3\n" \
      "s_branch LOOP1_%=\n" \
      "LSLOW0_%=:\n" \
      "v_mfma_f32_32x32x16_bf16 %0, v[160:163], v[200:203], %0\n" \
      "s_cmp_eq_u32 %5, 1\n" \
      "s_cbranch_scc0 LSK4_%=\n" \
      "s_add_u32 %7, %4, 1\n" \
      "v_readlane_b32 %6, v124, %7\n" \
      "s_add_u32 %7, %7, 1\n" \
      "s_lshr_b32 %8, %6, 16\n" \
      "s_cmp_eq_u32 %8, %7\n" \
      "s_cbranch_scc0 LSK4_%=\n" \
      "s_and_b32 %6, %6, 63\n" \
      "s_lshl_b32 %6, %6, 14\n" \
      "v_add_u32 %12, %6, %13\n" \
      "s_add_u32 %5, %5, 1\n" \
      "global_load_dwordx4 v[208:211], %12, %17 offset:0" XSC "\n" \
      "global_load_dwordx4 v[212:215], %12, %17 offset:1024" XSC "\n" \
      "global_load_dwordx4 v[216:219], %12, %17 offset:2048" XSC "\n" \
      "global_load_dwordx4 v[220:223], %12, %17 offset:3072" XSC "\n" \
      "LSK4_%=:\n" \
      "v_mfma_f32_32x32x16_bf16 %1, v[164:167], v[200:203], %1\n" \
      "s_cmp_eq_u32 %5, 2\n" \
      "s_cbranch_scc0 LSK5_%=\n" \
      "s_add_u32 %7, %4, 2\n" \
      "v_readlane_b32 %6, v124, %7\n" \
      "s_add_u32 %7, %7, 1\n" \
      "s_lshr_b32 %8, %6, 16\n" \
      "s_cmp_eq_u32 %8, %7\n" \
      "s_cbranch_scc0 LSK5_%=\n" \
      "s_and_b32 %6, %6, 63\n" \
      "s_lshl_b32 %6, %6, 14\n" \
      "v_add_u32 %12, %6, %13\n" \
      "s_add_u32 %5, %5, 1\n" \
      "global_load_dwordx4 v[224:227], %12, %17 offset:0" XSC "\n" \
      "global_load_dwordx4 v[228:231], %12, %17 offset:1024" XSC "\n" \
      "global_load_dwordx4 v[232:235], %12, %17 offset:2048" XSC "\n" \
      "global_load_dwordx4 v[236:239], %12, %17 offset:3072" XSC "\n" \
      "LSK5_%=:\n" \
      "v_mfma_f32_32x32x16_bf16 %2, v[168:171], v[200:203], %2\n" \
      "s_cmp_eq_u32 %5, 3\n" \
      "s_cbranch_scc0 LSK6_%=\n" \
      "s_add_u32 %7, %4, 3\n" \
      "v_readlane_b32 %6, v124, %7\n" \
      "s_add_u32 %7, %7, 1\n" \
      "s_lshr_b32 %8, %6, 16\n" \
      "s_cmp_eq_u32 %8, %7\n" \
      "s_cbranch_scc0 LSK6_%=\n" \
      "s_and_b32 %6, %6, 63\n" \
      "s_lshl_b32 %6, %6, 14\n" \
      "v_add_u32 %12, %6, %13\n" \
      "s_add_u32 %5, %5, 1\n" \
      "global_load_dwordx4 v[240:243], %12, %17 offset:0" XSC "\n" \
      "global_load_dwordx4 v[244:247], %12, %17 offset:1024" XSC "\n" \
      "global_load_dwordx4 v[248:251], %12, %17 offset:2048" XSC "\n" \
      "global_load_dwordx4 v[252:255], %12, %17 offset:3072" XSC "\n" \
      "LSK6_%=:\n" \
      "v_mfma_f32_32x32x16_bf16 %3, v[172:175], v[200:203], %3\n" \
      "v_mfma_f32_32x32x16_bf16 %0, v[176:179], v[204:207], %0\n" \
      "v_mfma_f32_32x32x16_bf16 %1, v[180:183], v[204:207], %1\n" \
      "v_mfma_f32_32x32x16_bf16 %2, v[184:187], v[204:207], %2\n" \
      "v_mfma_f32_32x32x16_bf16 %3, v[188:191], v[204:207], %3\n" \
      "s_branch LSPIN0_%=\n" \
      "LZZ0_%=:\n" \
      "s_add_u32 %9, %9, 1\n" \
      "s_bitcmp1_b32 %9, 26\n" \
      "s_cbranch_scc1 LEND_%=\n" \
      "s_sleep " CSLP "\n" \
      "LSPIN0_%=:\n" \
      "ds_read_b64 v[124:125], %15\n" \
      "s_waitcnt lgkmcnt(0)\n" \
      "s_cmp_eq_u32 %5, 1\n" \
      "s_cbranch_scc0 LSK7_%=\n" \
      "s_add_u32 %7, %4, 1\n" \
      "v_readlane_b32 %6, v124, %7\n" \
      "s_add_u32 %7, %7, 1\n" \
      "s_lshr_b32 %8, %6, 16\n" \
      "s_cmp_eq_u32 %8, %7\n" \
      "s_cbranch_scc0 LSK7_%=\n" \
      "s_and_b32 %6, %6, 63\n" \
      "s_lshl_b32 %6, %6, 14\n" \
      "v_add_u32 %12, %6, %13\n" \
      "s_add_u32 %5, %5, 1\n" \
      "global_load_dwordx4 v[208:211], %12, %17 offset:0" XSC "\n" \
      "global_load_dwordx4 v[212:215], %12, %17 offset:1024" XSC "\n" \
      "global_load_dwordx4 v[216:219], %12, %17 offset:2048" XSC "\n" \
      "global_load_dwordx4 v[220:223], %12, %17 offset:3072" XSC "\n" \
      "LSK7_%=:\n" \
      "s_cmp_eq_u32 %5, 2\n" \
      "s_cbranch_scc0 LSK8_%=\n" \
      "s_add_u32 %7, %4, 2\n" \
      "v_readlane_b32 %6, v124, %7\n" \
      "s_add_u32 %7, %7, 1\n" \
      "s_lshr_b32 %8, %6, 16\n" \
      "s_cmp_eq_u32 %8, %7\n" \
      "s_cbranch_scc0 LSK8_%=\n" \
      "s_and_b32 %6, %6, 63\n" \
      "s_lshl_b32 %6, %6, 14\n" \
      "v_add_u32 %12, %6, %13\n" \
      "s_add_u32 %5, %5, 1\n" \
      "global_load_dwordx4 v[224:227], %12, %17 offset:0" XSC "\n" \
      "global_load_dwordx4 v[228:231], %12, %17 offset:1024" XSC "\n" \
      "global_load_dwordx4 v[232:235], %12, %17 offset:2048" XSC "\n" \
      "global_load_dwordx4 v[236:239], %12, %17 offset:3072" XSC "\n" \
      "LSK8_%=:\n" \
      "s_cmp_eq_u32 %5, 3\n" \
      "s_cbranch_scc0 LSK9_%=\n" \
      "s_add_u32 %7, %4, 3\n" \
      "v_readlane_b32 %6, v124, %7\n" \
      "s_add_u32 %7, %7, 1\n" \
      "s_lshr_b32 %8, %6, 16\n" \
      "s_cmp_eq_u32 %8, %7\n" \
      "s_cbranch_scc0 LSK9_%=\n" \
      "s_and_b32 %6, %6, 63\n" \
      "s_lshl_b32 %6, %6, 14\n" \
      "v_add_u32 %12, %6, %13\n" \
      "s_add_u32 %5, %5, 1\n" \
      "global_load_dwordx4 v[240:243], %12, %17 offset:0" XSC "\n" \
      "global_load_dwordx4 v[244:247], %12, %17 offset:1024" XSC "\n" \
      "global_load_dwordx4 v[248:251], %12, %17 offset:2048" XSC "\n" \
      "global_load_dwordx4 v[252:255], %12, %17 offset:3072" XSC "\n" \
      "LSK9_%=:\n" \
      "s_add_u32 %7, %4, 1\n" \
      "v_readlane_b32 %6, v124, %7\n" \
      "s_add_u32 %7, %7, 1\n" \
      "s_lshr_b32 %8, %6, 16\n" \
      "s_cmp_eq_u32 %8, %7\n" \
      "s_cbranch_scc0 LZZ0_%=\n" \
      "s_bfe_u32 %6, %6, 0x60008\n" \
      "v_readlane_b32 %8, v125, %6\n" \
      "s_lshl_b32 %10, %6, 2\n" \
      "s_and_b32 %6, %6, 7\n" \
      "s_lshl_b32 %6, %6, 14\n" \
      "s_cmp_lg_u32 %8, 0\n" \
      "s_cbranch_scc0 LZZ0_%=\n" \
      "v_add_u32 %11, %6, %14\n" \
      "ds_read_b128 v[128:131], %11 offset:0\n" \
      "ds_read_b128 v[132:135], %11 offset:1024\n" \
      "ds_read_b128 v[136:139], %11 offset:2048\n" \
      "ds_read_b128 v[140:143], %11 offset:3072\n" \
      "ds_read_b128 v[144:147], %11 offset:4096\n" \
      "ds_read_b128 v[148:151], %11 offset:5120\n" \
      "ds_read_b128 v[152:155], %11 offset:6144\n" \
      "ds_read_b128 v[156:159], %11 offset:7168\n" \
      "LOOP1_%=:\n" \
      "s_add_u32 %4, %4, 1\n" \
      "s_sub_u32 %5, %5, 1\n" \
      "v_add_u32 %12, %10, %16\n" \
      "s_cmp_eq_u32 %5, 3\n" \
      "s_cbranch_scc1 LW8_1_%=\n" \
      "s_cmp_eq_u32 %5, 2\n" \
      "s_cbranch_scc1 LW4_1_%=\n" \
      "s_waitcnt vmcnt(0)\n" \
      "LW4_1_%=:\n" \
      "s_waitcnt vmcnt(4)\n" \
      "LW8_1_%=:\n" \
      "s_waitcnt vmcnt(8)\n" \
      "s_waitcnt lgkmcnt(7)\n" \
      "v_mfma_f32_32x32x16_bf16 %0, v[128:131], v[208:211], %0\n" \
      "ds_read_b128 v[160:163], %11 offset:8192\n" \
      "ds_read_b128 v[164:167], %11 offset:9216\n" \
      "s_waitcnt lgkmcnt(8)\n" \
      "v_mfma_f32_32x32x16_bf16 %1, v[132:135], v[208:211], %1\n" \
      "ds_read_b128 v[168:171], %11 offset:10240\n" \
      "ds_read_b128 v[172:175], %11 offset:11264\n" \
      "s_waitcnt lgkmcnt(9)\n" \
      "v_mfma_f32_32x32x16_bf16 %2, v[136:139], v[208:211], %2\n" \
      "ds_read_b128 v[176:179], %11 offset:12288\n" \
      "ds_read_b128 v[180:183], %11 offset:13312\n" \
      "s_waitcnt lgkmcnt(10)\n" \
      "v_mfma_f32_32x32x16_bf16 %3, v[140:143], v[208:211], %3\n" \
      "ds_read_b128 v[184:187], %11 offset:14336\n" \
      "ds_read_b128 v[188:191], %11 offset:15360\n" \
      "s_waitcnt lgkmcnt(8)\n" \
      "v_mfma_f32_32x32x16_bf16 %0, v[144:147], v[212:215], %0\n" \
      "ds_read_b64 v[124:125], %15\n" \
      "v_mfma_f32_32x32x16_bf16 %1, v[148:151], v[212:215], %1\n" \
      "v_mfma_f32_32x32x16_bf16 %2, v[152:155], v[212:215], %2\n" \
      "s_waitcnt lgkmcnt(0)\n" \
      "ds_write_b8 %12, %18\n" \
      "v_mfma_f32_32x32x16_bf16 %3, v[156:159], v[212:215], %3\n" \
      "s_add_u32 %7, %4, 1\n" \
      "v_readlane_b32 %6, v124, %7\n" \
      "s_add_u32 %7, %7, 1\n" \
      "s_lshr_b32 %8, %6, 16\n" \
      "s_cmp_eq_u32 %8, %7\n" \
      "s_cbranch_scc0 LSLOW1_%=\n" \
      "s_bfe_u32 %6, %6, 0x60008\n" \
      "v_readlane_b32 %8, v125, %6\n" \
      "s_lshl_b32 %10, %6, 2\n" \
      "s_and_b32 %6, %6, 7\n" \
      "s_lshl_b32 %6, %6, 14\n" \
      "s_cmp_lg_u32 %8, 0\n" \
      "s_cbranch_scc0 LSLOW1_%=\n" \
      "v_add_u32 %11, %6, %14\n" \
      "v_mfma_f32_32x32x16_bf16 %0, v[160:163], v[216:219], %0\n" \
      "ds_read_b128 v[128:131], %11 offset:0\n" \
      "ds_read_b128 v[132:135], %11 offset:1024\n" \
      "v_mfma_f32_32x32x16_bf16 %1, v[164:167], v[216:219], %1\n" \
      "ds_read_b128 v[136:139], %11 offset:2048\n" \
      "ds_read_b128 v[140:143], %11 offset:3072\n" \
      "v_mfma_f32_32x32x16_bf16 %2, v[168:171], v[216:219], %2\n" \
      "ds_read_b128 v[144:147], %11 offset:4096\n" \
      "ds_read_b128 v[148:151], %11 offset:5120\n" \
      "v_mfma_f32_32x32x16_bf16 %3, v[172:175], v[216:219], %3\n" \
      "ds_read_b128 v[152:155], %11 offset:6144\n" \
      "ds_read_b128 v[156:159], %11 offset:7168\n" \
      "v_mfma_f32_32x32x16_bf16 %0, v[176:179], v[220:223], %0\n" \
      "s_cmp_eq_u32 %5, 1\n" \
      "s_cbranch_scc0 LSK10_%=\n" \
      "s_add_u32 %7, %4, 1\n" \
      "v_readlane_b32 %6, v124, %7\n" \
      "s_add_u32 %7, %7, 1\n" \
      "s_lshr_b32 %8, %6, 16\n" \
      "s_cmp_eq_u32 %8, %7\n" \
      "s_cbranch_scc0 LSK10_%=\n" \
      "s_and_b32 %6, %6, 63\n" \
      "s_lshl_b32 %6, %6, 14\n" \
      "v_add_u32 %12, %6, %13\n" \
      "s_add_u32 %5, %5, 1\n" \
      "global_load_dwordx4 v[224:227], %12, %17 offset:0" XSC "\n" \
      "global_load_dwordx4 v[228:231], %12, %17 offset:1024" XSC "\n" \
      "global_load_dwordx4 v[232:235], %12, %17 offset:2048" XSC "\n" \
      "global_load_dwordx4 v[236:239], %12, %17 offset:3072" XSC "\n" \
      "LSK10_%=:\n" \
      "v_mfma_f32_32x32x16_bf16 %1, v[180:183], v[220:223], %1\n" \
      "s_cmp_eq_u32 %5, 2\n" \
      "s_cbranch_scc0 LSK11_%=\n" \
      "s_add_u32 %7, %4, 2\n" \
      "v_readlane_b32 %6, v124, %7\n" \
      "s_add_u32 %7, %7, 1\n" \
      "s_lshr_b32 %8, %6, 16\n" \
      "s_cmp_eq_u32 %8, %7\n" \
      "s_cbranch_scc0 LSK11_%=\n" \
      "s_and_b32 %6, %6, 63\n" \
      "s_lshl_b32 %6, %6, 14\n" \
      "v_add_u32 %12, %6, %13\n" \
      "s_add_u32 %5, %5, 1\n" \
      "global_load_dwordx4 v[240:243], %12, %17 offset:0" XSC "\n" \
      "global_load_dwordx4 v[244:247], %12, %17 offset:1024" XSC "\n" \
      "global_load_dwordx4 v[248:251], %12, %17 offset:2048" XSC "\n" \
      "global_load_dwordx4 v[252:255], %12, %17 offset:3072" XSC "\n" \
      "LSK11_%=:\n" \
      "v_mfma_f32_32x32x16_bf16 %2, v[184:187], v[220:223], %2\n" \
      "s_cmp_eq_u32 %5, 3\n" \
      "s_cbranch_scc0 LSK12_%=\n" \
      "s_add_u32 %7, %4, 3\n" \
      "v_readlane_b32 %6, v124, %7\n" \
      "s_add_u32 %7, %7, 1\n" \
      "s_lshr_b32 %8, %6, 16\n" \
      "s_cmp_eq_u32 %8, %7\n" \
      "s_cbranch_scc0 LSK12_%=\n" \
      "s_and_b32 %6, %6, 63\n" \
      "s_lshl_b32 %6, %6, 14\n" \
      "v_add_u32 %12, %6, %13\n" \
      "s_add_u32 %5, %5, 1\n" \
      "global_load_dwordx4 v[192:195], %12, %17 offset:0" XSC "\n" \
      "global_load_dwordx4 v[196:199], %12, %17 offset:1024" XSC "\n" \
      "global_load_dwordx4 v[200:203], %12, %17 offset:2048" XSC "\n" \
      "global_load_dwordx4 v[204:207], %12, %17 offset:3072" XSC "\n" \
      "LSK12_%=:\n" \
      "v_mfma_f32_32x32x16_bf16 %3, v[188:191], v[220:223], %3\n" \
      "s_branch LOOP2_%=\n" \
      "LSLOW1_%=:\n" \
      "v_mfma_f32_32x32x16_bf16 %0, v[160:163], v[216:219], %0\n" \
      "s_cmp_eq_u32 %5, 1\n" \
      "s_cbranch_scc0 LSK13_%=\n" \
      "s_add_u32 %7, %4, 1\n" \
      "v_readlane_b32 %6, v124, %7\n" \
      "s_add_u32 %7, %7, 1\n" \
      "s_lshr_b32 %8, %6, 16\n" \
      "s_cmp_eq_u32 %8, %7\n" \
      "s_cbranch_scc0 LSK13_%=\n" \
      "s_and_b32 %6, %6, 63\n" \
      "s_lshl_b32 %6, %6, 14\n" \
      "v_add_u32 %12, %6, %13\n" \
      "s_add_u32 %5, %5, 1\n" \
      "global_load_dwordx4 v[224:227], %12, %17 offset:0" XSC "\n" \
      "global_load_dwordx4 v[228:231], %12, %17 offset:1024" XSC "\n" \
      "global_load_dwordx4 v[232:235], %12, %17 offset:2048" XSC "\n" \
      "global_load_dwordx4 v[236:239], %12, %17 offset:3072" XSC "\n" \
      "LSK13_%=:\n" \
      "v_mfma_f32_32x32x16_bf16 %1, v[164:167], v[216:219], %1\n" \
      "s_cmp_eq_u32 %5, 2\n" \
      "s_cbranch_scc0 LSK14_%=\n" \
      "s_add_u32 %7, %4, 2\n" \
      "v_readlane_b32 %6, v124, %7\n" \
      "s_add_u32 %7, %7, 1\n" \
      "s_lshr_b32 %8, %6, 16\n" \
      "s_cmp_eq_u32 %8, %7\n" \
      "s_cbranch_scc0 LSK14_%=\n" \
      "s_and_b32 %6, %6, 63\n" \
      "s_lshl_b32 %6, %6, 14\n" \
      "v_add_u32 %12, %6, %13\n" \
      "s_add_u32 %5, %5, 1\n" \
      "global_load_dwordx4 v[240:243], %12, %17 offset:0" XSC "\n" \
      "global_load_dwordx4 v[244:247], %12, %17 offset:1024" XSC "\n" \
      "global_load_dwordx4 v[248:251], %12, %17 offset:2048" XSC "\n" \
      "global_load_dwordx4 v[252:255], %12, %17 offset:3072" XSC "\n" \
      "LSK14_%=:\n" \
      "v_mfma_f32_32x32x16_bf16 %2, v[168:171], v[216:219], %2\n" \
      "s_cmp_eq_u32 %5, 3\n" \
      "s_cbranch_scc0 LSK15_%=\n" \
      "s_add_u32 %7, %4, 3\n" \
      "v_readlane_b32 %6, v124, %7\n" \
      "s_add_u32 %7, %7, 1\n" \
      "s_lshr_b32 %8, %6, 16\n" \
      "s_cmp_eq_u32 %8, %7\n" \
      "s_cbranch_scc0 LSK15_%=\n" \
      "s_and_b32 %6, %6, 63\n" \
      "s_lshl_b32 %6, %6, 14\n" \
      "v_add_u32 %12, %6, %13\n" \
      "s_add_u32 %5, %5, 1\n" \
      "global_load_dwordx4 v[192:195], %12, %17 offset:0" XSC "\n" \
      "global_load_dwordx4 v[196:199], %12, %17 offset:1024" XSC "\n" \
      "global_load_dwordx4 v[200:203], %12, %17 offset:2048" XSC "\n" \
      "global_load_dwordx4 v[204:207], %12, %17 offset:3072" XSC "\n" \
      "LSK15_%=:\n" \
      "v_mfma_f32_32x32x16_bf16 %3, v[172:175], v[216:219], %3\n" \
      "v_mfma_f32_32x32x16_bf16 %0, v[176:179], v[220:223], %0\n" \
      "v_mfma_f32_32x32x16_bf16 %1, v[180:183], v[220:223], %1\n" \
      "v_mfma_f32_32x32x16_bf16 %2, v[184:187], v[220:223], %2\n" \
      "v_mfma_f32_32x32x16_bf16 %3, v[188:191], v[220:223], %3\n" \
      "s_branch LSPIN1_%=\n" \
      "LZZ1_%=:\n" \
      "s_add_u32 %9, %9, 1\n" \
      "s_bitcmp1_b32 %9, 26\n" \
      "s_cbranch_scc1 LEND_%=\n" \
      "s_sleep " CSLP "\n" \
      "LSPIN1_%=:\n" \
      "ds_read_b64 v[124:125], %15\n" \
      "s_waitcnt lgkmcnt(0)\n" \
      "s_cmp_eq_u32 %5, 1\n" \
      "s_cbranch_scc0 LSK16_%=\n" \
      "s_add_u32 %7, %4, 1\n" \
      "v_readlane_b32 %6, v124, %7\n" \
      "s_add_u32 %7, %7, 1\n" \
      "s_lshr_b32 %8, %6, 16\n" \
      "s_cmp_eq_u32 %8, %7\n" \
      "s_cbranch_scc0 LSK16_%=\n" \
      "s_and_b32 %6, %6, 63\n" \
      "s_lshl_b32 %6, %6, 14\n" \
      "v_add_u32 %12, %6, %13\n" \
      "s_add_u32 %5, %5, 1\n" \
      "global_load_dwordx4 v[224:227], %12, %17 offset:0" XSC "\n" \
      "global_load_dwordx4 v[228:231], %12, %17 offset:1024" XSC "\n" \
      "global_load_dwordx4 v[232:235], %12, %17 offset:2048" XSC "\n" \
      "global_load_dwordx4 v[236:239], %12, %17 offset:3072" XSC "\n" \
      "LSK16_%=:\n" \
      "s_cmp_eq_u32 %5, 2\n" \
      "s_cbranch_scc0 LSK17_%=\n" \
      "s_add_u32 %7, %4, 2\n" \
      "v_readlane_b32 %6, v124, %7\n" \
      "s_add_u32 %7, %7, 1\n" \
      "s_lshr_b32 %8, %6, 16\n" \
      "s_cmp_eq_u32 %8, %7\n" \
      "s_cbranch_scc0 LSK17_%=\n" \
      "s_and_b32 %6, %6, 63\n" \
      "s_lshl_b32 %6, %6, 14\n" \
      "v_add_u32 %12, %6, %13\n" \
      "s_add_u32 %5, %5, 1\n" \
      "global_load_dwordx4 v[240:243], %12, %17 offset:0" XSC "\n" \
      "global_load_dwordx4 v[244:247], %12, %17 offset:1024" XSC "\n" \
      "global_load_dwordx4 v[248:251], %12, %17 offset:2048" XSC "\n" \
      "global_load_dwordx4 v[252:255], %12, %17 offset:3072" XSC "\n" \
      "LSK17_%=:\n" \
      "s_cmp_eq_u32 %5, 3\n" \
      "s_cbranch_scc0 LSK18_%=\n" \
      "s_add_u32 %7, %4, 3\n" \
      "v_readlane_b32 %6, v124, %7\n" \
      "s_add_u32 %7, %7, 1\n" \
      "s_lshr_b32 %8, %6, 16\n" \
      "s_cmp_eq_u32 %8, %7\n" \
      "s_cbranch_scc0 LSK18_%=\n" \
      "s_and_b32 %6, %6, 63\n" \
      "s_lshl_b32 %6, %6, 14\n" \
      "v_add_u32 %12, %6, %13\n" \
      "s_add_u32 %5, %5, 1\n" \
      "global_load_dwordx4 v[192:195], %12, %17 offset:0" XSC "\n" \
      "global_load_dwordx4 v[196:199], %12, %17 offset:1024" XSC "\n" \
      "global_load_dwordx4 v[200:203], %12, %17 offset:2048" XSC "\n" \
      "global_load_dwordx4 v[204:207], %12, %17 offset:3072" XSC "\n" \
      "LSK18_%=:\n" \
      "s_add_u32 %7, %4, 1\n" \
      "v_readlane_b32 %6, v124, %7\n" \
      "s_add_u32 %7, %7, 1\n" \
      "s_lshr_b32 %8, %6, 16\n" \
      "s_cmp_eq_u32 %8, %7\n" \
      "s_cbranch_scc0 LZZ1_%=\n" \
      "s_bfe_u32 %6, %6, 0x60008\n" \
      "v_readlane_b32 %8, v125, %6\n" \
      "s_lshl_b32 %10, %6, 2\n" \
      "s_and_b32 %6, %6, 7\n" \
      "s_lshl_b32 %6, %6, 14\n" \
      "s_cmp_lg_u32 %8, 0\n" \
      "s_cbranch_scc0 LZZ1_%=\n" \
      "v_add_u32 %11, %6, %14\n" \
      "ds_read_b128 v[128:131], %11 offset:0\n" \
      "ds_read_b128 v[132:135], %11 offset:1024\n" \
      "ds_read_b128 v[136:139], %11 offset:2048\n" \
      "ds_read_b128 v[140:143], %11 offset:3072\n" \
      "ds_read_b128 v[144:147], %11 offset:4096\n" \
      "ds_read_b128 v[148:151], %11 offset:5120\n" \
      "ds_read_b128 v[152:155], %11 offset:6144\n" \
      "ds_read_b128 v[156:159], %11 offset:7168\n" \
      "LOOP2_%=:\n" \
      "s_add_u32 %4, %4, 1\n" \
      "s_sub_u32 %5, %5, 1\n" \
      "v_add_u32 %12, %10, %16\n" \
      "s_cmp_eq_u32 %5, 3\n" \
      "s_cbranch_scc1 LW8_2_%=\n" \
      "s_cmp_eq_u32 %5, 2\n" \
      "s_cbranch_scc1 LW4_2_%=\n" \
      "s_waitcnt vmcnt(0)\n" \
      "LW4_2_%=:\n" \
      "s_waitcnt vmcnt(4)\n" \
      "LW8_2_%=:\n" \
      "s_waitcnt vmcnt(8)\n" \
      "s_waitcnt lgkmcnt(7)\n" \
      "v_mfma_f32_32x32x16_bf16 %0, v[128:131], v[224:227], %0\n" \
      "ds_read_b128 v[160:163], %11 offset:8192\n" \
      "ds_read_b128 v[164:167], %11 offset:9216\n" \
      "s_waitcnt lgkmcnt(8)\n" \
      "v_mfma_f32_32x32x16_bf16 %1, v[132:135], v[224:227], %1\n" \
      "ds_read_b128 v[168:171], %11 offset:10240\n" \
      "ds_read_b128 v[172:175], %11 offset:11264\n" \
      "s_waitcnt lgkmcnt(9)\n" \
      "v_mfma_f32_32x32x16_bf16 %2, v[136:139], v[224:227], %2\n" \
      "ds_read_b128 v[176:179], %11 offset:12288\n" \
      "ds_read_b128 v[180:183], %11 offset:13312\n" \
      "s_waitcnt lgkmcnt(10)\n" \
      "v_mfma_f32_32x32x16_bf16 %3, v[140:143], v[224:227], %3\n" \
      "ds_read_b128 v[184:187], %11 offset:14336\n" \
      "ds_read_b128 v[188:191], %11 offset:15360\n" \
      "s_waitcnt lgkmcnt(8)\n" \
      "v_mfma_f32_32x32x16_bf16 %0, v[144:147], v[228:231], %0\n" \
      "ds_read_b64 v[124:125], %15\n" \
      "v_mfma_f32_32x32x16_bf16 %1, v[148:151], v[228:231], %1\n" \
      "v_mfma_f32_32x32x16_bf16 %2, v[152:155], v[228:231], %2\n" \
      "s_waitcnt lgkmcnt(0)\n" \
      "ds_write_b8 %12, %18\n" \
      "v_mfma_f32_32x32x16_bf16 %3, v[156:159], v[228:231], %3\n" \
      "s_add_u32 %7, %4, 1\n" \
      "v_readlane_b32 %6, v124, %7\n" \
      "s_add_u32 %7, %7, 1\n" \
      "s_lshr_b32 %8, %6, 16\n" \
      "s_cmp_eq_u32 %8, %7\n" \
      "s_cbranch_scc0 LSLOW2_%=\n" \
      "s_bfe_u32 %6, %6, 0x60008\n" \
      "v_readlane_b32 %8, v125, %6\n" \
      "s_lshl_b32 %10, %6, 2\n" \
      "s_and_b32 %6, %6, 7\n" \
      "s_lshl_b32 %6, %6, 14\n" \
      "s_cmp_lg_u32 %8, 0\n" \
      "s_cbranch_scc0 LSLOW2_%=\n" \
      "v_add_u32 %11, %6, %14\n" \
      "v_mfma_f32_32x32x16_bf16 %0, v[160:163], v[232:235], %0\n" \
      "ds_read_b128 v[128:131], %11 offset:0\n" \
      "ds_read_b128 v[132:135], %11 offset:1024\n" \
      "v_mfma_f32_32x32x16_bf16 %1, v[164:167], v[232:235], %1\n" \
      "ds_read_b128 v[136:139], %11 offset:2048\n" \
      "ds_read_b128 v[140:143], %11 offset:3072\n" \
      "v_mfma_f32_32x32x16_bf16 %2, v[168:171], v[232:235], %2\n" \
      "ds_read_b128 v[144:147], %11 offset:4096\n" \
      "ds_read_b128 v[148:151], %11 offset:5120\n" \
      "v_mfma_f32_32x32x16_bf16 %3, v[172:175], v[232:235], %3\n" \
      "ds_read_b128 v[152:155], %11 offset:6144\n" \
      "ds_read_b128 v[156:159], %11 offset:7168\n" \
      "v_mfma_f32_32x32x16_bf16 %0, v[176:179], v[236:239], %0\n" \
      "s_cmp_eq_u32 %5, 1\n" \
      "s_cbranch_scc0 LSK19_%=\n" \
      "s_add_u32 %7, %4, 1\n" \
      "v_readlane_b32 %6, v124, %7\n" \
      "s_add_u32 %7, %7, 1\n" \
      "s_lshr_b32 %8, %6, 16\n" \
      "s_cmp_eq_u32 %8, %7\n" \
      "s_cbranch_scc0 LSK19_%=\n" \
      "s_and_b32 %6, %6, 63\n" \
      "s_lshl_b32 %6, %6, 14\n" \
      "v_add_u32 %12, %6, %13\n" \
      "s_add_u32 %5, %5, 1\n" \
      "global_load_dwordx4 v[240:243], %12, %17 offset:0" XSC "\n" \
      "global_load_dwordx4 v[244:247], %12, %17 offset:1024" XSC "\n" \
      "global_load_dwordx4 v[248:251], %12, %17 offset:2048" XSC "\n" \
      "global_load_dwordx4 v[252:255], %12, %17 offset:3072" XSC "\n" \
      "LSK19_%=:\n" \
      "v_mfma_f32_32x32x16_bf16 %1, v[180:183], v[236:239], %1\n" \
      "s_cmp_eq_u32 %5, 2\n" \
      "s_cbranch_scc0 LSK20_%=\n" \
      "s_add_u32 %7, %4, 2\n" \
      "v_readlane_b32 %6, v124, %7\n" \
      "s_add_u32 %7, %7, 1\n" \
      "s_lshr_b32 %8, %6, 16\n" \
      "s_cmp_eq_u32 %8, %7\n" \
      "s_cbranch_scc0 LSK20_%=\n" \
      "s_and_b32 %6, %6, 63\n" \
      "s_lshl_b32 %6, %6, 14\n" \
      "v_add_u32 %12, %6, %13\n" \
      "s_add_u32 %5, %5, 1\n" \
      "global_load_dwordx4 v[192:195], %12, %17 offset:0" XSC "\n" \
      "global_load_dwordx4 v[196:199], %12, %17 offset:1024" XSC "\n" \
      "global_load_dwordx4 v[200:203], %12, %17 offset:2048" XSC "\n" \
      "global_load_dwordx4 v[204:207], %12, %17 offset:3072" XSC "\n" \
      "LSK20_%=:\n" \
      "v_mfma_f32_32x32x16_bf16 %2, v[184:187], v[236:239], %2\n" \
      "s_cmp_eq_u32 %5, 3\n" \
      "s_cbranch_scc0 LSK21_%=\n" \
      "s_add_u32 %7, %4, 3\n" \
      "v_readlane_b32 %6, v124, %7\n" \
      "s_add_u32 %7, %7, 1\n" \
      "s_lshr_b32 %8, %6, 16\n" \
      "s_cmp_eq_u32 %8, %7\n" \
      "s_cbranch_scc0 LSK21_%=\n" \
      "s_and_b32 %6, %6, 63\n" \
      "s_lshl_b32 %6, %6, 14\n" \
      "v_add_u32 %12, %6, %13\n" \
      "s_add_u32 %5, %5, 1\n" \
      "global_load_dwordx4 v[208:211], %12, %17 offset:0" XSC "\n" \
      "global_load_dwordx4 v[212:215], %12, %17 offset:1024" XSC "\n" \
      "global_load_dwordx4 v[216:219], %12, %17 offset:2048" XSC "\n" \
      "global_load_dwordx4 v[220:223], %12, %17 offset:3072" XSC "\n" \
      "LSK21_%=:\n" \
      "v_mfma_f32_32x32x16_bf16 %3, v[188:191], v[236:239], %3\n" \
      "s_branch LOOP3_%=\n" \
      "LSLOW2_%=:\n" \
      "v_mfma_f32_32x32x16_bf16 %0, v[160:163], v[232:235], %0\n" \
      "s_cmp_eq_u32 %5, 1\n" \
      "s_cbranch_scc0 LSK22_%=\n" \
      "s_add_u32 %7, %4, 1\n" \
      "v_readlane_b32 %6, v124, %7\n" \
      "s_add_u32 %7, %7, 1\n" \
      "s_lshr_b32 %8, %6, 16\n" \
      "s_cmp_eq_u32 %8, %7\n" \
      "s_cbranch_scc0 LSK22_%=\n" \
      "s_and_b32 %6, %6, 63\n" \
      "s_lshl_b32 %6, %6, 14\n" \
      "v_add_u32 %12, %6, %13\n" \
      "s_add_u32 %5, %5, 1\n" \
      "global_load_dwordx4 v[240:243], %12, %17 offset:0" XSC "\n" \
      "global_load_dwordx4 v[244:247], %12, %17 offset:1024" XSC "\n" \
      "global_load_dwordx4 v[248:251], %12, %17 offset:2048" XSC "\n" \
      "global_load_dwordx4 v[252:255], %12, %17 offset:3072" XSC "\n" \
      "LSK22_%=:\n" \
      "v_mfma_f32_32x32x16_bf16 %1, v[164:167], v[232:235], %1\n" \
      "s_cmp_eq_u32 %5, 2\n" \
      "s_cbranch_scc0 LSK23_%=\n" \
      "s_add_u32 %7, %4, 2\n" \
      "v_readlane_b32 %6, v124, %7\n" \
      "s_add_u32 %7, %7, 1\n" \
      "s_lshr_b32 %8, %6, 16\n" \
      "s_cmp_eq_u32 %8, %7\n" \
      "s_cbranch_scc0 LSK23_%=\n" \
      "s_and_b32 %6, %6, 63\n" \
      "s_lshl_b32 %6, %6, 14\n" \
      "v_add_u32 %12, %6, %13\n" \
      "s_add_u32 %5, %5, 1\n" \
      "global_load_dwordx4 v[192:195], %12, %17 offset:0" XSC "\n" \
      "global_load_dwordx4 v[196:199], %12, %17 offset:1024" XSC "\n" \
      "global_load_dwordx4 v[200:203], %12, %17 offset:2048" XSC "\n" \
      "global_load_dwordx4 v[204:207], %12, %17 offset:3072" XSC "\n" \
      "LSK23_%=:\n" \
      "v_mfma_f32_32x32x16_bf16 %2, v[168:171], v[232:235], %2\n" \
      "s_cmp_eq_u32 %5, 3\n" \
      "s_cbranch_scc0 LSK24_%=\n" \
      "s_add_u32 %7, %4, 3\n" \
      "v_readlane_b32 %6, v124, %7\n" \
      "s_add_u32 %7, %7, 1\n" \
      "s_lshr_b32 %8, %6, 16\n" \
      "s_cmp_eq_u32 %8, %7\n" \
      "s_cbranch_scc0 LSK24_%=\n" \
      "s_and_b32 %6, %6, 63\n" \
      "s_lshl_b32 %6, %6, 14\n" \
      "v_add_u32 %12, %6, %13\n" \
      "s_add_u32 %5, %5, 1\n" \
      "global_load_dwordx4 v[208:211], %12, %17 offset:0" XSC "\n" \
      "global_load_dwordx4 v[212:215], %12, %17 offset:1024" XSC "\n" \
      "global_load_dwordx4 v[216:219], %12, %17 offset:2048" XSC "\n" \
      "global_load_dwordx4 v[220:223], %12, %17 offset:3072" XSC "\n" \
      "LSK24_%=:\n" \
      "v_mfma_f32_32x32x16_bf16 %3, v[172:175], v[232:235], %3\n" \
      "v_mfma_f32_32x32x16_bf16 %0, v[176:179], v[236:239], %0\n" \
      "v_mfma_f32_32x32x16_bf16 %1, v[180:183], v[236:239], %1\n" \
      "v_mfma_f32_32x32x16_bf16 %2, v[184:187], v[236:239], %2\n" \
      "v_mfma_f32_32x32x16_bf16 %3, v[188:191], v[236:239], %3\n" \
      "s_branch LSPIN2_%=\n" \
      "LZZ2_%=:\n" \
      "s_add_u32 %9, %9, 1\n" \
      "s_bitcmp1_b32 %9, 26\n" \
      "s_cbranch_scc1 LEND_%=\n" \
      "s_sleep " CSLP "\n" \
      "LSPIN2_%=:\n" \
      "ds_read_b64 v[124:125], %15\n" \
      "s_waitcnt lgkmcnt(0)\n" \
      "s_cmp_eq_u32 %5, 1\n" \
      "s_cbranch_scc0 LSK25_%=\n" \
      "s_add_u32 %7, %4, 1\n" \
      "v_readlane_b32 %6, v124, %7\n" \
      "s_add_u32 %7, %7, 1\n" \
      "s_lshr_b32 %8, %6, 16\n" \
      "s_cmp_eq_u32 %8, %7\n" \
      "s_cbranch_scc0 LSK25_%=\n" \
      "s_and_b32 %6, %6, 63\n" \
      "s_lshl_b32 %6, %6, 14\n" \
      "v_add_u32 %12, %6, %13\n" \
      "s_add_u32 %5, %5, 1\n" \
      "global_load_dwordx4 v[240:243], %12, %17 offset:0" XSC "\n" \
      "global_load_dwordx4 v[244:247], %12, %17 offset:1024" XSC "\n" \
      "global_load_dwordx4 v[248:251], %12, %17 offset:2048" XSC "\n" \
      "global_load_dwordx4 v[252:255], %12, %17 offset:3072" XSC "\n" \
      "LSK25_%=:\n" \
      "s_cmp_eq_u32 %5, 2\n" \
      "s_cbranch_scc0 LSK26_%=\n" \
      "s_add_u32 %7, %4, 2\n" \
      "v_readlane_b32 %6, v124, %7\n" \
      "s_add_u32 %7, %7, 1\n" \
      "s_lshr_b32 %8, %6, 16\n" \
      "s_cmp_eq_u32 %8, %7\n" \
      "s_cbranch_scc0 LSK26_%=\n" \
      "s_and_b32 %6, %6, 63\n" \
      "s_lshl_b32 %6, %6, 14\n" \
      "v_add_u32 %12, %6, %13\n" \
      "s_add_u32 %5, %5, 1\n" \
      "global_load_dwordx4 v[192:195], %12, %17 offset:0" XSC "\n" \
      "global_load_dwordx4 v[196:199], %12, %17 offset:1024" XSC "\n" \
      "global_load_dwordx4 v[200:203], %12, %17 offset:2048" XSC "\n" \
      "global_load_dwordx4 v[204:207], %12, %17 offset:3072" XSC "\n" \
      "LSK26_%=:\n" \
      "s_cmp_eq_u32 %5, 3\n" \
      "s_cbranch_scc0 LSK27_%=\n" \
      "s_add_u32 %7, %4, 3\n" \
      "v_readlane_b32 %6, v124, %7\n" \
      "s_add_u32 %7, %7, 1\n" \
      "s_lshr_b32 %8, %6, 16\n" \
      "s_cmp_eq_u32 %8, %7\n" \
      "s_cbranch_scc0 LSK27_%=\n" \
      "s_and_b32 %6, %6, 63\n" \
      "s_lshl_b32 %6, %6, 14\n" \
      "v_add_u32 %12, %6, %13\n" \
      "s_add_u32 %5, %5, 1\n" \
      "global_load_dwordx4 v[208:211], %12, %17 offset:0" XSC "\n" \
      "global_load_dwordx4 v[212:215], %12, %17 offset:1024" XSC "\n" \
      "global_load_dwordx4 v[216:219], %12, %17 offset:2048" XSC "\n" \
      "global_load_dwordx4 v[220:223], %12, %17 offset:3072" XSC "\n" \
      "LSK27_%=:\n" \
      "s_add_u32 %7, %4, 1\n" \
      "v_readlane_b32 %6, v124, %7\n" \
      "s_add_u32 %7, %7, 1\n" \
      "s_lshr_b32 %8, %6, 16\n" \
      "s_cmp_eq_u32 %8, %7\n" \
      "s_cbranch_scc0 LZZ2_%=\n" \
      "s_bfe_u32 %6, %6, 0x60008\n" \
      "v_readlane_b32 %8, v125, %6\n" \
      "s_lshl_b32 %10, %6, 2\n" \
      "s_and_b32 %6, %6, 7\n" \
      "s_lshl_b32 %6, %6, 14\n" \
      "s_cmp_lg_u32 %8, 0\n" \
      "s_cbranch_scc0 LZZ2_%=\n" \
      "v_add_u32 %11, %6, %14\n" \
      "ds_read_b128 v[128:131], %11 offset:0\n" \
      "ds_read_b128 v[132:135], %11 offset:1024\n" \
      "ds_read_b128 v[136:139], %11 offset:2048\n" \
      "ds_read_b128 v[140:143], %11 offset:3072\n" \
      "ds_read_b128 v[144:147], %11 offset:4096\n" \
      "ds_read_b128 v[148:151], %11 offset:5120\n" \
      "ds_read_b128 v[152:155], %11 offset:6144\n" \
      "ds_read_b128 v[156:159], %11 offset:7168\n" \
      "LOOP3_%=:\n" \
      "s_add_u32 %4, %4, 1\n" \
      "s_sub_u32 %5, %5, 1\n" \
      "v_add_u32 %12, %10, %16\n" \
      "s_cmp_eq_u32 %5, 3\n" \
      "s_cbranch_scc1 LW8_3_%=\n" \
      "s_cmp_eq_u32 %5, 2\n" \
      "s_cbranch_scc1 LW4_3_%=\n" \
      "s_waitcnt vmcnt(0)\n" \
      "LW4_3_%=:\n" \
      "s_waitcnt vmcnt(4)\n" \
      "LW8_3_%=:\n" \
      "s_waitcnt vmcnt(8)\n" \
      "s_waitcnt lgkmcnt(7)\n" \
      "v_mfma_f32_32x32x16_bf16 %0, v[128:131], v[240:243], %0\n" \
      "ds_read_b128 v[160:163], %11 offset:8192\n" \
      "ds_read_b128 v[164:167], %11 offset:9216\n" \
      "s_waitcnt lgkmcnt(8)\n" \
      "v_mfma_f32_32x32x16_bf16 %1, v[132:135], v[240:243], %1\n" \
      "ds_read_b128 v[168:171], %11 offset:10240\n" \
      "ds_read_b128 v[172:175], %11 offset:11264\n" \
      "s_waitcnt lgkmcnt(9)\n" \
      "v_mfma_f32_32x32x16_bf16 %2, v[136:139], v[240:243], %2\n" \
      "ds_read_b128 v[176:179], %11 offset:12288\n" \
      "ds_read_b128 v[180:183], %11 offset:13312\n" \
      "s_waitcnt lgkmcnt(10)\n" \
      "v_mfma_f32_32x32x16_bf16 %3, v[140:143], v[240:243], %3\n" \
      "ds_read_b128 v[184:187], %11 offset:14336\n" \
      "ds_read_b128 v[188:191], %11 offset:15360\n" \
      "s_waitcnt lgkmcnt(8)\n" \
      "v_mfma_f32_32x32x16_bf16 %0, v[144:147], v[244:247], %0\n" \
      "ds_read_b64 v[124:125], %15\n" \
      "v_mfma_f32_32x32x16_bf16 %1, v[148:151], v[244:247], %1\n" \
      "v_mfma_f32_32x32x16_bf16 %2, v[152:155], v[244:247], %2\n" \
      "s_waitcnt lgkmcnt(0)\n" \
      "ds_write_b8 %12, %18\n" \
      "v_mfma_f32_32x32x16_bf16 %3, v[156:159], v[244:247], %3\n" \
      "s_add_u32 %7, %4, 1\n" \
      "v_readlane_b32 %6, v124, %7\n" \
      "s_add_u32 %7, %7, 1\n" \
      "s_lshr_b32 %8, %6, 16\n" \
      "s_cmp_eq_u32 %8, %7\n" \
      "s_cbranch_scc0 LSLOW3_%=\n" \
      "s_bfe_u32 %6, %6, 0x60008\n" \
      "v_readlane_b32 %8, v125, %6\n" \
      "s_lshl_b32 %10, %6, 2\n" \
      "s_and_b32 %6, %6, 7\n" \
      "s_lshl_b32 %6, %6, 14\n" \
      "s_cmp_lg_u32 %8, 0\n" \
      "s_cbranch_scc0 LSLOW3_%=\n" \
      "v_add_u32 %11, %6, %14\n" \
      "v_mfma_f32_32x32x16_bf16 %0, v[160:163], v[248:251], %0\n" \
      "ds_read_b128 v[128:131], %11 offset:0\n" \
      "ds_read_b128 v[132:135], %11 offset:1024\n" \
      "v_mfma_f32_32x32x16_bf16 %1, v[164:167], v[248:251], %1\n" \
      "ds_read_b128 v[136:139], %11 offset:2048\n" \
      "ds_read_b128 v[140:143], %11 offset:3072\n" \
      "v_mfma_f32_32x32x16_bf16 %2, v[168:171], v[248:251], %2\n" \
      "ds_read_b128 v[144:147], %11 offset:4096\n" \
      "ds_read_b128 v[148:151], %11 offset:5120\n" \
      "v_mfma_f32_32x32x16_bf16 %3, v[172:175], v[248:251], %3\n" \
      "ds_read_b128 v[152:155], %11 offset:6144\n" \
      "ds_read_b128 v[156:159], %11 offset:7168\n" \
      "v_mfma_f32_32x32x16_bf16 %0, v[176:179], v[252:255], %0\n" \
      "s_cmp_eq_u32 %5, 1\n" \
      "s_cbranch_scc0 LSK28_%=\n" \
      "s_add_u32 %7, %4, 1\n" \
      "v_readlane_b32 %6, v124, %7\n" \
      "s_add_u32 %7, %7, 1\n" \
      "s_lshr_b32 %8, %6, 16\n" \
      "s_cmp_eq_u32 %8, %7\n" \
      "s_cbranch_scc0 LSK28_%=\n" \
      "s_and_b32 %6, %6, 63\n" \
      "s_lshl_b32 %6, %6, 14\n" \
      "v_add_u32 %12, %6, %13\n" \
      "s_add_u32 %5, %5, 1\n" \
      "global_load_dwordx4 v[192:195], %12, %17 offset:0" XSC "\n" \
      "global_load_dwordx4 v[196:199], %12, %17 offset:1024" XSC "\n" \
      "global_load_dwordx4 v[200:203], %12, %17 offset:2048" XSC "\n" \
      "global_load_dwordx4 v[204:207], %12, %17 offset:3072" XSC "\n" \
      "LSK28_%=:\n" \
      "v_mfma_f32_32x32x16_bf16 %1, v[180:183], v[252:255], %1\n" \
      "s_cmp_eq_u32 %5, 2\n" \
      "s_cbranch_scc0 LSK29_%=\n" \
      "s_add_u32 %7, %4, 2\n" \
      "v_readlane_b32 %6, v124, %7\n" \
      "s_add_u32 %7, %7, 1\n" \
      "s_lshr_b32 %8, %6, 16\n" \
      "s_cmp_eq_u32 %8, %7\n" \
      "s_cbranch_scc0 LSK29_%=\n" \
      "s_and_b32 %6, %6, 63\n" \
      "s_lshl_b32 %6, %6, 14\n" \
      "v_add_u32 %12, %6, %13\n" \
      "s_add_u32 %5, %5, 1\n" \
      "global_load_dwordx4 v[208:211], %12, %17 offset:0" XSC "\n" \
      "global_load_dwordx4 v[212:215], %12, %17 offset:1024" XSC "\n" \
      "global_load_dwordx4 v[216:219], %12, %17 offset:2048" XSC "\n" \
      "global_load_dwordx4 v[220:223], %12, %17 offset:3072" XSC "\n" \
      "LSK29_%=:\n" \
      "v_mfma_f32_32x32x16_bf16 %2, v[184:187], v[252:255], %2\n" \
      "s_cmp_eq_u32 %5, 3\n" \
      "s_cbranch_scc0 LSK30_%=\n" \
      "s_add_u32 %7, %4, 3\n" \
      "v_readlane_b32 %6, v124, %7\n" \
      "s_add_u32 %7, %7, 1\n" \
      "s_lshr_b32 %8, %6, 16\n" \
      "s_cmp_eq_u32 %8, %7\n" \
      "s_cbranch_scc0 LSK30_%=\n" \
      "s_and_b32 %6, %6, 63\n" \
      "s_lshl_b32 %6, %6, 14\n" \
      "v_add_u32 %12, %6, %13\n" \
      "s_add_u32 %5, %5, 1\n" \
      "global_load_dwordx4 v[224:227], %12, %17 offset:0" XSC "\n" \
      "global_load_dwordx4 v[228:231], %12, %17 offset:1024" XSC "\n" \
      "global_load_dwordx4 v[232:235], %12, %17 offset:2048" XSC "\n" \
      "global_load_dwordx4 v[236:239], %12, %17 offset:3072" XSC "\n" \
      "LSK30_%=:\n" \
      "v_mfma_f32_32x32x16_bf16 %3, v[188:191], v[252:255], %3\n" \
      "s_branch LOOP0_%=\n" \
      "LSLOW3_%=:\n" \
      "v_mfma_f32_32x32x16_bf16 %0, v[160:163], v[248:251], %0\n" \
      "s_cmp_eq_u32 %5, 1\n" \
      "s_cbranch_scc0 LSK31_%=\n" \
      "s_add_u32 %7, %4, 1\n" \
      "v_readlane_b32 %6, v124, %7\n" \
      "s_add_u32 %7, %7, 1\n" \
      "s_lshr_b32 %8, %6, 16\n" \
      "s_cmp_eq_u32 %8, %7\n" \
      "s_cbranch_scc0 LSK31_%=\n" \
      "s_and_b32 %6, %6, 63\n" \
      "s_lshl_b32 %6, %6, 14\n" \
      "v_add_u32 %12, %6, %13\n" \
      "s_add_u32 %5, %5, 1\n" \
      "global_load_dwordx4 v[192:195], %12, %17 offset:0" XSC "\n" \
      "global_load_dwordx4 v[196:199], %12, %17 offset:1024" XSC "\n" \
      "global_load_dwordx4 v[200:203], %12, %17 offset:2048" XSC "\n" \
      "global_load_dwordx4 v[204:207], %12, %17 offset:3072" XSC "\n" \
      "LSK31_%=:\n" \
      "v_mfma_f32_32x32x16_bf16 %1, v[164:167], v[248:251], %1\n" \
      "s_cmp_eq_u32 %5, 2\n" \
      "s_cbranch_scc0 LSK32_%=\n" \
      "s_add_u32 %7, %4, 2\n" \
      "v_readlane_b32 %6, v124, %7\n" \
      "s_add_u32 %7, %7, 1\n" \
      "s_lshr_b32 %8, %6, 16\n" \
      "s_cmp_eq_u32 %8, %7\n" \
      "s_cbranch_scc0 LSK32_%=\n" \
      "s_and_b32 %6, %6, 63\n" \
      "s_lshl_b32 %6, %6, 14\n" \
      "v_add_u32 %12, %6, %13\n" \
      "s_add_u32 %5, %5, 1\n" \
      "global_load_dwordx4 v[208:211], %12, %17 offset:0" XSC "\n" \
      "global_load_dwordx4 v[212:215], %12, %17 offset:1024" XSC "\n" \
      "global_load_dwordx4 v[216:219], %12, %17 offset:2048" XSC "\n" \
      "global_load_dwordx4 v[220:223], %12, %17 offset:3072" XSC "\n" \
      "LSK32_%=:\n" \
      "v_mfma_f32_32x32x16_bf16 %2, v[168:171], v[248:251], %2\n" \
      "s_cmp_eq_u32 %5, 3\n" \
      "s_cbranch_scc0 LSK33_%=\n" \
      "s_add_u32 %7, %4, 3\n" \
      "v_readlane_b32 %6, v124, %7\n" \
      "s_add_u32 %7, %7, 1\n" \
      "s_lshr_b32 %8, %6, 16\n" \
      "s_cmp_eq_u32 %8, %7\n" \
      "s_cbranch_scc0 LSK33_%=\n" \
      "s_and_b32 %6, %6, 63\n" \
      "s_lshl_b32 %6, %6, 14\n" \
      "v_add_u32 %12, %6, %13\n" \
      "s_add_u32 %5, %5, 1\n" \
      "global_load_dwordx4 v[224:227], %12, %17 offset:0" XSC "\n" \
      "global_load_dwordx4 v[228:231], %12, %17 offset:1024" XSC "\n" \
      "global_load_dwordx4 v[232:235], %12, %17 offset:2048" XSC "\n" \
      "global_load_dwordx4 v[236:239], %12, %17 offset:3072" XSC "\n" \
      "LSK33_%=:\n" \
      "v_mfma_f32_32x32x16_bf16 %3, v[172:175], v[248:251], %3\n" \
      "v_mfma_f32_32x32x16_bf16 %0, v[176:179], v[252:255], %0\n" \
      "v_mfma_f32_32x32x16_bf16 %1, v[180:183], v[252:255], %1\n" \
      "v_mfma_f32_32x32x16_bf16 %2, v[184:187], v[252:255], %2\n" \
      "v_mfma_f32_32x32x16_bf16 %3, v[188:191], v[252:255], %3\n" \
      "s_cmp_eq_u32 %4, 63\n" \
      "s_cbranch_scc1 LEND_%=\n" \
      "s_branch LSPIN3_%=\n" \
      "LZZ3_%=:\n" \
      "s_add_u32 %9, %9, 1\n" \
      "s_bitcmp1_b32 %9, 26\n" \
      "s_cbranch_scc1 LEND_%=\n" \
      "s_sleep " CSLP "\n" \
      "LSPIN3_%=:\n" \
      "ds_read_b64 v[124:125], %15\n" \
      "s_waitcnt lgkmcnt(0)\n" \
      "s_cmp_eq_u32 %5, 1\n" \
      "s_cbranch_scc0 LSK34_%=\n" \
      "s_add_u32 %7, %4, 1\n" \
      "v_readlane_b32 %6, v124, %7\n" \
      "s_add_u32 %7, %7, 1\n" \
      "s_lshr_b32 %8, %6, 16\n" \
      "s_cmp_eq_u32 %8, %7\n" \
      "s_cbranch_scc0 LSK34_%=\n" \
      "s_and_b32 %6, %6, 63\n" \
      "s_lshl_b32 %6, %6, 14\n" \
      "v_add_u32 %12, %6, %13\n" \
      "s_add_u32 %5, %5, 1\n" \
      "global_load_dwordx4 v[192:195], %12, %17 offset:0" XSC "\n" \
      "global_load_dwordx4 v[196:199], %12, %17 offset:1024" XSC "\n" \
      "global_load_dwordx4 v[200:203], %12, %17 offset:2048" XSC "\n" \
      "global_load_dwordx4 v[204:207], %12, %17 offset:3072" XSC "\n" \
      "LSK34_%=:\n" \
      "s_cmp_eq_u32 %5, 2\n" \
      "s_cbranch_scc0 LSK35_%=\n" \
      "s_add_u32 %7, %4, 2\n" \
      "v_readlane_b32 %6, v124, %7\n" \
      "s_add_u32 %7, %7, 1\n" \
      "s_lshr_b32 %8, %6, 16\n" \
      "s_cmp_eq_u32 %8, %7\n" \
      "s_cbranch_scc0 LSK35_%=\n" \
      "s_and_b32 %6, %6, 63\n" \
      "s_lshl_b32 %6, %6, 14\n" \
      "v_add_u32 %12, %6, %13\n" \
      "s_add_u32 %5, %5, 1\n" \
      "global_load_dwordx4 v[208:211], %12, %17 offset:0" XSC "\n" \
      "global_load_dwordx4 v[212:215], %12, %17 offset:1024" XSC "\n" \
      "global_load_dwordx4 v[216:219], %12, %17 offset:2048" XSC "\n" \
      "global_load_dwordx4 v[220:223], %12, %17 offset:3072" XSC "\n" \
      "LSK35_%=:\n" \
      "s_cmp_eq_u32 %5, 3\n" \
      "s_cbranch_scc0 LSK36_%=\n" \
      "s_add_u32 %7, %4, 3\n" \
      "v_readlane_b32 %6, v124, %7\n" \
      "s_add_u32 %7, %7, 1\n" \
      "s_lshr_b32 %8, %6, 16\n" \
      "s_cmp_eq_u32 %8, %7\n" \
      "s_cbranch_scc0 LSK36_%=\n" \
      "s_and_b32 %6, %6, 63\n" \
      "s_lshl_b32 %6, %6, 14\n" \
      "v_add_u32 %12, %6, %13\n" \
      "s_add_u32 %5, %5, 1\n" \
      "global_load_dwordx4 v[224:227], %12, %17 offset:0" XSC "\n" \
      "global_load_dwordx4 v[228:231], %12, %17 offset:1024" XSC "\n" \
      "global_load_dwordx4 v[232:235], %12, %17 offset:2048" XSC "\n" \
      "global_load_dwordx4 v[236:239], %12, %17 offset:3072" XSC "\n" \
      "LSK36_%=:\n" \
      "s_add_u32 %7, %4, 1\n" \
      "v_readlane_b32 %6, v124, %7\n" \
      "s_add_u32 %7, %7, 1\n" \
      "s_lshr_b32 %8, %6, 16\n" \
      "s_cmp_eq_u32 %8, %7\n" \
      "s_cbranch_scc0 LZZ3_%=\n" \
      "s_bfe_u32 %6, %6, 0x60008\n" \
      "v_readlane_b32 %8, v125, %6\n" \
      "s_lshl_b32 %10, %6, 2\n" \
      "s_and_b32 %6, %6, 7\n" \
      "s_lshl_b32 %6, %6, 14\n" \
      "s_cmp_lg_u32 %8, 0\n" \
      "s_cbranch_scc0 LZZ3_%=\n" \
      "v_add_u32 %11, %6, %14\n" \
      "ds_read_b128 v[128:131], %11 offset:0\n" \
      "ds_read_b128 v[132:135], %11 offset:1024\n" \
      "ds_read_b128 v[136:139], %11 offset:2048\n" \
      "ds_read_b128 v[140:143], %11 offset:3072\n" \
      "ds_read_b128 v[144:147], %11 offset:4096\n" \
      "ds_read_b128 v[148:151], %11 offset:5120\n" \
      "ds_read_b128 v[152:155], %11 offset:6144\n" \
      "ds_read_b128 v[156:159], %11 offset:7168\n" \
      "s_branch LOOP0_%=\n" \
      "LEND_%=:\n" \
      "s_waitcnt vmcnt(0) lgkmcnt(0)\n" \
      "s_nop 7\n" \
      "s_nop 7\n"
#define CONSUME_CLOB "v124", "v125", "v128", "v129", "v130", "v131", "v132", "v133", "v134", "v135", "v136", "v137", "v138", "v139", "v140", "v141", "v142", "v143", "v144", "v145", "v146", "v147", "v148", "v149", "v150", "v151", "v152", "v153", "v154", "v155", "v156", "v157", "v158", "v159", "v160", "v161", "v162", "v163", "v164", "v165", "v166", "v167", "v168", "v169", "v170", "v171", "v172", "v173", "v174", "v175", "v176", "v177", "v178", "v179", "v180", "v181", "v182", "v183", "v184", "v185", "v186", "v187", "v188", "v189", "v190", "v191", "v192", "v193", "v194", "v195", "v196", "v197", "v198", "v199", "v200", "v201", "v202", "v203", "v204", "v205", "v206", "v207", "v208", "v209", "v210", "v211", "v212", "v213", "v214", "v215", "v216", "v217", "v218", "v219", "v220", "v221", "v222", "v223", "v224", "v225", "v226", "v227", "v228", "v229", "v230", "v231", "v232", "v233", "v234", "v235", "v236", "v237", "v238", "v239", "v240", "v241", "v242", "v243", "v244", "v245", "v246", "v247", "v248", "v249", "v250", "v251", "v252", "v253", "v254", "v255"

#define ORDER_TABLE(X) \
  X(0, 0xf18800) X(1, 0xe412c40) X(2, 0x4f28810) X(3, 0x12404c50) X(4, 0x1d38818) X(5, 0x16422c58) \
  X(6, 0x5648898) X(7, 0x8f32cd8) X(8, 0x5d58808) X(9, 0x1a414c48) X(10, 0x9d01088) X(11, 0xcf688c8) \
  X(12, 0x1e478820) X(13, 0xdd11060) X(14, 0x224888a0) X(15, 0x26406ce0) X(16, 0x2a403120) X(17, 0x2e498960) \
  X(18, 0x1716da0) X(19, 0x57a89e0) X(20, 0x2513220) X(21, 0x6526e60) X(22, 0xa5b8aa0) X(23, 0x1e36ee0) \
  X(24, 0xe523019) X(25, 0x5e05001) X(26, 0x12533059) X(27, 0x9e01411) X(28, 0x16511441) X(29, 0xde08c99) \
  X(30, 0x1a5150d9) X(31, 0x1e518c09) X(32, 0x22503421) X(33, 0x26528c49) X(34, 0x2a513451) X(35, 0x2e538c61) \
  X(36, 0x2648c89) X(37, 0x1f234a1) X(38, 0x6658ce1) X(39, 0x5f334c9) X(40, 0xa668d21) X(41, 0x9f07002) \
  X(42, 0xe678d61) X(43, 0xdf019a1) X(44, 0x12605442) X(45, 0x166171e1) X(46, 0x1a688c0a) X(47, 0x1e627012) \
  X(48, 0x22611a21) X(49, 0x26698e61) X(50, 0x2a63704a) X(51, 0x2e603aa1) X(52, 0x27a8c1a) X(53, 0x671548a) \
  X(54, 0xa7b8ee1) X(55, 0xe71385a) X(56, 0x12723852) X(57, 0x167338ca) X(58, 0x1a701c9a) X(59, 0x1e7090da) \
  X(60, 0x22719003) X(61, 0x26711c43) X(62, 0x2a705822) X(63, 0x2e72900b)

constexpr unsigned M83 = 0x83838383u, M1C = 0x1c1c1c1cu, M03 = 0x03030303u, M80 = 0x80808080u, M7C = 0x7c7c7c7cu;

__device__ __forceinline__ unsigned mean_top(unsigned mx) { return rdl(wave_scan(mx >> 16), 63) >> 6; }
__device__ __forceinline__ unsigned encode(const u32x4 (&f)[8], unsigned mean, char* ws, unsigned su, LAS char* sc, unsigned lane) {
  const unsigned vo = lane * 16;
  const unsigned top = (mean + 12) >> 7;
  const unsigned e0 = (top > 7 ? top - 7 : 0) & 255;
  const unsigned base = e0 * 0x00800080u;
  u32x4 lo[4], xp[4];
  unsigned H[16], hp[6], msk[2] = {0u, 0u};
#pragma unroll
  for (int y = 0; y < 16; ++y) {
    const unsigned t0 = pk_sub(f[y >> 1][2 * (y & 1)], base), t1 = pk_sub(f[y >> 1][2 * (y & 1) + 1], base);
    lo[y >> 2][y & 3] = perm(t1, t0, 0x06040200u);
    H[y] = perm(t1, t0, 0x07050301u);
  }
  char* e = ws + OFF_EXPI + su * IU;
  st16x4(e, vo, lo[0], lo[1], lo[2], lo[3]);
  __builtin_amdgcn_sched_barrier(0);
#pragma unroll
  for (int h = 0; h < 2; ++h) {
    const unsigned* G = H + 8 * h;
    hp[3 * h + 0] = bfi(M83, G[0], bfi(M1C, rotr(G[1], 29), G[6] << 5));
    hp[3 * h + 1] = bfi(M83, G[2], bfi(M1C, rotr(G[3], 29), bfi(0x20202020u, G[6] >> 2, G[7] >> 1)));
    hp[3 * h + 2] = bfi(M83, G[4], bfi(M1C, rotr(G[5], 29), G[7] << 5));
  }
  st16(e + 4096, vo, u32x4{hp[0], hp[1], hp[2], hp[3]});
  st8(e + 5120, lane * 8, u32x2{hp[4], hp[5]});
  __builtin_amdgcn_sched_barrier(0);
#pragma unroll
  for (int y = 0; y < 16; ++y) {
    const unsigned x = H[y] & M7C;
    xp[y >> 2][y & 3] = x;
    msk[y >> 3] |= ((x + M7C) >> (y & 7)) & (M80 >> (y & 7));
  }
#pragma unroll
  for (int q = 0; q < 4; ++q) *(LAS u32x4*)(sc + q * 1024 + vo) = xp[q];
  const unsigned cnt = __builtin_popcount(msk[0]) + __builtin_popcount(msk[1]);
  const unsigned inc = wave_scan(cnt);
  const unsigned tot = rdl(inc, 63);
  unsigned ent[2] = {0u, 0u};
  bool bad = tot > 127;
  if (!bad) {
    unsigned slot = inc - cnt + 1;
    u64 m = ((u64)msk[1] << 32) | msk[0];
    while (m) {
      *(LAS unsigned short*)(sc + 4096 + slot * 2) = (unsigned short)(lane * 64 + __builtin_ctzll(m));
      m &= m - 1;
      ++slot;
    }
    CFENCE();
    const unsigned codes = *(const LAS unsigned*)(sc + 4096 + lane * 4);
    const unsigned x5z = ((256 - e0) & 255) >> 3;
    bool lb = false;
#pragma unroll
    for (int s = 0; s < 2; ++s) {
      const unsigned idx = 2 * lane + s, code = (codes >> (16 * s)) & 4095;
      const unsigned y = ((code >> 2) & 8) + 7 - (code & 7);
      const unsigned pos = (y >> 2) * 1024 + (code >> 6) * 16 + (y & 3) * 4 + ((code >> 3) & 3);
      const unsigned x5 = (unsigned)*(const LAS unsigned char*)(sc + pos) >> 2;
      const bool on = idx >= 1 && idx <= tot;
      const bool direct = x5 < 8 || x5 > 24;
      lb = lb || (on && !direct && x5 != x5z);
      ent[s] = on ? (pos << 4) | (direct ? (x5 & 15) : 8u) : 0u;
    }
    bad = __ballot(lb) != 0;
  }
  if (bad) {
    ent[0] = ent[1] = 0;
    st16x4(ws + OFF_EXPP + su * 4096, vo, xp[0], xp[1], xp[2], xp[3]);
  }
  if (lane == 0) ent[0] = e0 | (bad ? 0x200u : (tot > 63 ? 0x100u : 0u));
  return ent[0] | (ent[1] << 16);
}
constexpr unsigned M07 = 0x07070707u, M01 = 0x01010101u, M0F = 0x0f0f0f0fu, M55 = 0x55555555u;
__device__ __forceinline__ unsigned mul24(unsigned a, unsigned b) {
  unsigned r;
  asm("v_mul_u32_u24 %0, %2, %1" : "=v"(r) : "v"(a), "s"(b));
  return r;
}
#define SB() __builtin_amdgcn_sched_barrier(0)
__device__ __forceinline__ unsigned sym_class(unsigned d, unsigned lhi, unsigned llo) {
  return perm(lhi, llo, d | ((d & 0x08080808u) << 1));
}
__device__ __forceinline__ void scatter4(unsigned r, unsigned e, unsigned& off) {
  unsigned a, t, u;
#define SD(b) " dst_sel:DWORD dst_unused:UNUSED_PAD src0_sel:BYTE_" #b " src1_sel:DWORD\n\t"
  asm volatile("v_mul_u32_u24 %0, %6, %5\n\tv_xor_b32 %1, %7, %5\n\tv_lshl_or_b32 %0, %0, 8, %1\n\t"
               "v_add_u32_sdwa %1, %0, %3" SD(0) "v_add_u32_sdwa %2, %0, %3" SD(2)
               "ds_write_b8 %1, %4\n\tds_write_b8_d16_hi %2, %4\n\t"
               "v_add_u32_sdwa %1, %0, %3" SD(1) "v_add_u32_sdwa %2, %0, %3" SD(3)
               "v_lshrrev_b32 %0, 8, %4\n\tv_bcnt_u32_b32 %3, %5, %3\n\t"
               "ds_write_b8 %1, %0\n\tds_write_b8_d16_hi %2, %0\n\tv_bcnt_u32_b32 %3, %5, %3"
               : "=&v"(a), "=&v"(t), "=&v"(u), "+v"(off) : "v"(r), "v"(e), "s"(0x020202u), "s"(M01) : "memory");
}
__device__ __forceinline__ unsigned select4(unsigned c, unsigned& off) {
  const unsigned m = perm(0u, 0xff000000u, c), e = m & M01;
  off += __builtin_popcount(e);
  return c + (((mul24(e, 0x010101u) << 8) + e) & m);
}
__device__ __forceinline__ unsigned lds32(unsigned a) { return *(const LAS unsigned*)(size_t)a; }
struct Pair { unsigned lo, hi; };
__device__ __forceinline__ Pair lds_pair(unsigned a) {
  const LAS unsigned* p = (const LAS unsigned*)(size_t)(a & ~3u);
  return Pair{p[0], p[1]};
}
__device__ __forceinline__ unsigned at(Pair p, unsigned a) { return __builtin_amdgcn_alignbyte(p.hi, p.lo, a); }
__device__ __forceinline__ unsigned esc_count(unsigned w) { return __builtin_popcount(w & (w >> 1) & M55); }
template <bool LIST>
__device__ __forceinline__ unsigned tenc(const unsigned (&d)[16], unsigned l1hi, unsigned l1lo, unsigned l2hi, unsigned l2lo,
                                         unsigned plain, unsigned top, unsigned sc, unsigned lane, u32x4& w1) {
#pragma unroll
  for (int g = 0; g < 16; ++g) {
    const unsigned c = sym_class(d[g], l1hi, l1lo);
    w1[g >> 2] = (g & 3) ? bfi(M03 << (2 * (g & 3)), c << (2 * (g & 3)), w1[g >> 2]) : c;
  }
  const unsigned cnt = esc_count(w1[0]) + esc_count(w1[1]) + esc_count(w1[2]) + esc_count(w1[3]);
  const unsigned inc = wave_scan(cnt), n2 = rdl(inc, 63);
  if (n2 > 2016) return top | 0x80000000u;
  unsigned off = sc + 2 * (inc - cnt);
#pragma unroll
  for (int q = 0; q < 4; ++q) {
    unsigned ew = w1[q] & (w1[q] >> 1);
    asm("" : "+v"(ew));
#pragma unroll
    for (int i = 0; i < 4; ++i) scatter4(d[4 * q + i], (ew >> (2 * i)) & M01, off);
  }
  if (lane < 16) *(LAS unsigned char*)(size_t)(sc + 2 * (n2 + lane)) = (unsigned char)plain;
  CFENCE();
  const unsigned d2 = (n2 + 15) >> 4, var = sc + 6144;
  unsigned n3 = 0, nl = 0;
#pragma unroll 1
  for (unsigned j0 = 0; j0 < d2; j0 += 64) {
    const unsigned j = j0 + lane;
    const u32x4 a = *(const LAS u32x4*)(size_t)(sc + j * 32), b = *(const LAS u32x4*)(size_t)(sc + j * 32 + 16);
    const unsigned r[4] = {perm(a[1], a[0], 0x06040200u), perm(a[3], a[2], 0x06040200u),
                           perm(b[1], b[0], 0x06040200u), perm(b[3], b[2], 0x06040200u)};
    unsigned w = 0;
#pragma unroll
    for (int i = 0; i < 4; ++i) {
      const unsigned c = sym_class(r[i], l2hi, l2lo);
      w = i ? bfi(M03 << (2 * i), c << (2 * i), w) : c;
    }
    w = j < d2 ? w : 0u;
    const unsigned c3 = esc_count(w), i3 = wave_scan(c3), t3 = rdl(i3, 63);
    if (n3 + t3 > 1008) return top | 0x80000000u;
    if (j < d2) *(LAS unsigned*)(size_t)(var + 4 + j * 4) = w;
    unsigned o3 = sc + 4096 + 2 * (n3 + i3 - c3);
#pragma unroll
    for (int i = 0; i < 4; ++i) scatter4(r[i], ((w & (w >> 1)) >> (2 * i)) & M01, o3);
    n3 += t3;
    if (LIST) {
      unsigned m = 0;
#pragma unroll
      for (int i = 0; i < 4; ++i) m |= ((((r[i] & 0x7f7f7f7fu) + 0x6a6a6a6au) | r[i]) & M80) >> i;
      m = j < d2 ? m : 0u;
      if (__ballot(m != 0)) {
        const unsigned cl = __builtin_popcount(m), il = wave_scan(cl), tl = rdl(il, 63);
        if (nl + tl > 63) return top | 0x80000000u;
        unsigned slot = var + 4 + (d2 + nl + il - cl) * 4;
        for (; m; m &= m - 1, slot += 4) {
          const unsigned bit = __builtin_ctz(m), i = 7 - (bit & 7), by = bit >> 3;
          const unsigned rr = (i & 2) ? ((i & 1) ? r[3] : r[2]) : ((i & 1) ? r[1] : r[0]);
          *(LAS unsigned*)(size_t)slot = ((j * 16 + i * 4 + by) << 8) | ((top - (rr >> (8 * by))) & 255);
        }
        nl += tl;
      }
    }
  }
  CFENCE();
  unsigned cells = sc + 4096;
  if (!LIST && LV4) {
    if (lane < 16) *(LAS unsigned char*)(size_t)(cells + 2 * (n3 + lane)) = 6;
    CFENCE();
    const u32x4 a = *(const LAS u32x4*)(size_t)(cells + lane * 32), b = *(const LAS u32x4*)(size_t)(cells + lane * 32 + 16);
    const unsigned r[4] = {perm(a[1], a[0], 0x06040200u), perm(a[3], a[2], 0x06040200u),
                           perm(b[1], b[0], 0x06040200u), perm(b[3], b[2], 0x06040200u)};
    unsigned w = 0;
#pragma unroll
    for (int i = 0; i < 4; ++i) {
      const unsigned c = sym_class(r[i] - 0x06060606u, l1hi, l1lo);
      w = i ? bfi(M03 << (2 * i), c << (2 * i), w) : c;
    }
    nl = (n3 + 15) >> 4;
    w = lane < nl ? w : 0u;
    const unsigned c4 = esc_count(w), i4 = wave_scan(c4);
    if (lane < nl) *(LAS unsigned*)(size_t)(var + 4 + (d2 + lane) * 4) = w;
    cells = sc;
    unsigned o4 = cells + 2 * (i4 - c4);
#pragma unroll
    for (int i = 0; i < 4; ++i) scatter4(r[i], ((w & (w >> 1)) >> (2 * i)) & M01, o4);
    n3 = rdl(i4, 63);
    CFENCE();
  }
  const unsigned d3 = (n3 + 7) >> 3;
  if (1 + d2 + nl + d3 > 256) return top | 0x80000000u;
#pragma unroll 1
  for (unsigned j0 = 0; j0 < d3; j0 += 64) {
    const unsigned j = j0 + lane;
    const u32x4 a = *(const LAS u32x4*)(size_t)(cells + j * 16);
    const unsigned x = (perm(a[1], a[0], 0x06040200u) & M0F) + 0x0a0a0a0au, y = (perm(a[3], a[2], 0x06040200u) & M0F) + 0x0a0a0a0au;
    if (j < d3) *(LAS unsigned*)(size_t)(var + 4 + (d2 + nl + j) * 4) = bfi(M0F, x, y << 4);
  }
  const unsigned hdr = top | (d2 << 8) | (d3 << 16) | (nl << 24);
  if (lane == 0) *(LAS unsigned*)(size_t)var = hdr;
  CFENCE();
  return hdr;
}
__device__ __forceinline__ unsigned tlines(unsigned hdr) {
  return (((hdr >> 8) & 255) + ((hdr >> 16) & 255) + ((hdr >> 24) & 63) + 32) >> 5;
}
template <bool SUB>
__device__ __forceinline__ void tdec(u32x4 w1, unsigned hdr, unsigned lut1, unsigned lut2, unsigned k3, unsigned sc,
                                     unsigned lane, unsigned (&x)[16]) {
  const unsigned d2 = (hdr >> 8) & 255, d3 = (hdr >> 16) & 255, nl = (hdr >> 24) & 63;
  const unsigned s3 = sc + 1024, s2 = sc + 2176;
  const unsigned wa = lds32(sc + ((1 + d2 + nl + lane) & 255) * 4), wb = lds32(sc + ((65 + d2 + nl + lane) & 255) * 4);
  unsigned w = lds32(sc + 4 + lane * 4);
  const unsigned w2 = lds32(sc + 260 + lane * 4), le = lds32(sc + (1 + d2 + lane) * 4);
  SB();
  const unsigned cnt = esc_count(w1[0]) + esc_count(w1[1]) + esc_count(w1[2]) + esc_count(w1[3]);
  unsigned off = s2 + wave_scan(cnt) - cnt;
  SB();
  const unsigned sn = (!SUB && LV4) ? s2 : s3;
  *(LAS u32x2*)(size_t)(sn + lane * 8) = SUB ? u32x2{k3 - (wa & M0F), k3 - ((wa >> 4) & M0F)} : u32x2{k3 + ((wa & M0F) << 3), k3 + ((wa >> 1) & 0x78787878u)};
  if (d3 > 64)
    *(LAS u32x2*)(size_t)(sn + 512 + lane * 8) =
        SUB ? u32x2{k3 - (wb & M0F), k3 - ((wb >> 4) & M0F)} : u32x2{k3 + ((wb & M0F) << 3), k3 + ((wb >> 1) & 0x78787878u)};
  CFENCE();
  if (!SUB && LV4) {
    const unsigned v = lane < nl ? le : 0u, c4 = esc_count(v);
    unsigned o4 = s2 + wave_scan(c4) - c4, s4[4], a4[4];
    Pair y[4];
#pragma unroll
    for (int i = 0; i < 4; ++i) {
      a4[i] = o4;
      y[i] = lds_pair(o4);
      s4[i] = select4((v >> (2 * i)) & M03, o4);
    }
    *(LAS u32x4*)(size_t)(s3 + lane * 16) =
        u32x4{perm(at(y[0], a4[0]), 0x403830u, s4[0]), perm(at(y[1], a4[1]), 0x403830u, s4[1]),
              perm(at(y[2], a4[2]), 0x403830u, s4[2]), perm(at(y[3], a4[3]), 0x403830u, s4[3])};
    CFENCE();
  }
  unsigned sel[16], ad[16];
  Pair xp[16];
  {
    w = lane < d2 ? w : 0u;
    const unsigned c3 = esc_count(w), i3 = wave_scan(c3);
    unsigned o3 = s3 + i3 - c3, s4[4], a4[4];
    Pair y[4];
#pragma unroll
    for (int i = 0; i < 4; ++i) {
      a4[i] = o3;
      y[i] = lds_pair(o3);
      s4[i] = select4((w >> (2 * i)) & M03, o3);
    }
    SB();
#pragma unroll
    for (int g = 0; g < 4; ++g) {
      ad[g] = off;
      sel[g] = select4((w1[0] >> (2 * g)) & M03, off);
    }
    SB();
    *(LAS u32x4*)(size_t)(s2 + lane * 16) =
        u32x4{perm(at(y[0], a4[0]), lut2, s4[0]), perm(at(y[1], a4[1]), lut2, s4[1]),
              perm(at(y[2], a4[2]), lut2, s4[2]), perm(at(y[3], a4[3]), lut2, s4[3])};
    if (d2 > 64) {
      const unsigned v = lane + 64 < d2 ? w2 : 0u, c4 = esc_count(v);
      o3 = s3 + rdl(i3, 63) + wave_scan(c4) - c4;
#pragma unroll
      for (int i = 0; i < 4; ++i) {
        a4[i] = o3;
        y[i] = lds_pair(o3);
        s4[i] = select4((v >> (2 * i)) & M03, o3);
      }
      *(LAS u32x4*)(size_t)(s2 + 1024 + lane * 16) =
          u32x4{perm(at(y[0], a4[0]), lut2, s4[0]), perm(at(y[1], a4[1]), lut2, s4[1]),
              perm(at(y[2], a4[2]), lut2, s4[2]), perm(at(y[3], a4[3]), lut2, s4[3])};
    }
  }
  if (SUB && lane < nl) *(LAS unsigned char*)(size_t)(s2 + ((le >> 8) & 2047)) = (unsigned char)le;
  CFENCE();
#pragma unroll
  for (int g = 0; g < 16; ++g) {
    if (g >= 4) {
      ad[g] = off;
      sel[g] = select4((w1[g >> 2] >> (2 * (g & 3))) & M03, off);
    }
    xp[g] = lds_pair(ad[g]);
    SB();
  }
#pragma unroll
  for (int g = 0; g < 16; ++g) x[g] = perm(at(xp[g], ad[g]), lut1, sel[g]);
  CFENCE();
}
__device__ __forceinline__ void encode2(const u32x4 (&f)[8], unsigned mean, char* ws, unsigned su, unsigned sc, unsigned lane) {
  const unsigned vo = lane * 16;
  unsigned top = (mean + 30) >> 7;
  top = top < 21 ? 21 : (top > 255 ? 255 : top);
  const unsigned k1 = top * 0x01000100u + 0x00ff00ffu;
  u32x4 sm[4];
  unsigned d[16];
#pragma unroll
  for (int g = 0; g < 16; ++g) {
    const unsigned a = f[g >> 1][2 * (g & 1)], b = f[g >> 1][2 * (g & 1) + 1];
    sm[g >> 2][g & 3] = bfi(M80, perm(b, a, 0x07050301u), perm(b, a, 0x06040200u));
    unsigned ta, tb;
    asm("v_pk_mad_u16 %0, %1, %2, %3" : "=v"(ta) : "v"(a), "s"(0xfffefffeu), "v"(k1));
    asm("v_pk_mad_u16 %0, %1, %2, %3" : "=v"(tb) : "v"(b), "s"(0xfffefffeu), "v"(k1));
    d[g] = perm(tb, ta, 0x07050301u);
  }
  char* e = ws + OFF_EXPI + su * IU;
  st16x4(e, vo, sm[0], sm[1], sm[2], sm[3]);
  u32x4 w1;
  const unsigned hdr = tenc<true>(d, 0x03030303u, 0x02010003u, 0x03030201u, 0x03030300u, 0u, top, sc, lane, w1);
  if (hdr >> 31) {
    st16x4(ws + OFF_EXPP + su * 4096, vo, u32x4{d[0], d[1], d[2], d[3]}, u32x4{d[4], d[5], d[6], d[7]},
           u32x4{d[8], d[9], d[10], d[11]}, u32x4{d[12], d[13], d[14], d[15]});
    if (lane == 0) st4(e + 5120, 0u, hdr);
  } else {
    st16(e + 4096, vo, w1);
    const u32x4 v = *(const LAS u32x4*)(size_t)(sc + 6144 + vo);
    if (lane < 8 * tlines(hdr)) st16(e + 5120, vo, v);
  }
}
__device__ __forceinline__ unsigned smear(unsigned d) { return perm(d, d << 8, 0x0b090a08u); }
__device__ __forceinline__ unsigned quant(float y, float inv) {
  return f2u(__builtin_amdgcn_fmed3f(__builtin_fmaf(y, inv, 12582912.0f), 12582785.0f, 12583039.0f));
}
__device__ __forceinline__ unsigned pack4(unsigned b0, unsigned b1, unsigned b2, unsigned b3) {
  return perm(perm(b3, b2, 0x04000400u), perm(b1, b0, 0x04000400u), 0x05040100u);
}
__device__ __forceinline__ unsigned out_offset(unsigned p, unsigned m) {
  const unsigned z = (m >> 2) & 127, tau = z < 32 ? (z & 3) : (z >> 3), u = z < 32 ? (z >> 2) : (z & 7);
  return ((p >> 2) * 2048 + (((tau & 3) << 2) | (tau >> 2)) * 128 + (m & 3) * 32) * NN + (p & 3) * 1024 + u * 128;
}
template <int SLEEP>
__device__ __forceinline__ unsigned wait_amax(char* ws, unsigned wg, unsigned lane, unsigned call, float& inv) {
  char* ga = lane < 8 ? ws + OFF_AMAX2 + ((wg & (NREP - 1)) * 8 + lane) * 128 : ws + OFF_TMO;
  const u64 t0 = rt();
  unsigned state, m;
  for (;;) {
    u32x2 a;
    asm volatile("s_mov_b64 exec, 0x1ff\n\tglobal_load_dwordx2 %0, %1, off" SC "\n\ts_mov_b64 exec, -1\n\ts_waitcnt vmcnt(0)"
                 : "=&v"(a) : "v"(ga) : "memory");
    m = lane < 8 ? a[0] : 0u;
    const unsigned have = (unsigned)__ballot(a[1] >= call) & 0xff;
    if (have == 0xff) { state = 1; break; }
    if (give_up(ws, rdl(a[0], 8), call, t0)) { state = 2; break; }
    if (SLEEP) __builtin_amdgcn_s_sleep(SLEEP);
    if (ASL0 && have == 0) __builtin_amdgcn_s_sleep(ASL0);
  }
  const float amax = u2f(wave_max(m));
  inv = amax > 0.f ? 127.0f / amax : 0.f;
  return state;
}

__global__ void __launch_bounds__(NTHR, 2) fused(Params P) {
  extern __shared__ __attribute__((aligned(16))) char smem[];
  const float* px = P.x;
  const float* pw = P.w;
  char* pout = P.out;
  char* ws = P.ws;
  char* wsa[NR];
#pragma unroll
  for (int q = 0; q < NR; ++q) wsa[q] = P.ws_all[q];
  unsigned rank = P.rank;
  const unsigned wg = blockIdx.x ^ 128;
  const unsigned lane = threadIdx.x & 63;
  const unsigned wave = rfl(threadIdx.x >> 6);
  const bool is_gemm = wg < NGEMM;
  const unsigned lds0 = (unsigned)(size_t)(LAS char*)smem;

  const unsigned gw = wg & 127;
  unsigned role = 6 | (wg & 127) << 8;
  if (is_gemm || wave != 1) {
    const unsigned lidx = (gw - 84) * 6 + (wave == 0 ? 0 : wave - 2);
    const unsigned wc = (wg & 7) < 2 ? (wg >> 3) * 2 + (wg & 7) : 30 + (wg >> 3) * 6 + (wg & 7);
    const unsigned xj = (wc - 32) * 2 + wave - 6;
    if (is_gemm) {
      role = (wc >= 32 && wave >= 1 && (wave <= 5 || wc < 112) ? 2u : 0u) | wc << 16 |
             (wave >= 6 ? (((1 + (xj >> 5)) & 7) << 4) | (xj & 31) << 8 : wave << 4 | wc << 8);
    } else {
      role = (gw >= 84 && wave != 2 && lidx < 256 ? 3u | (6 + (lidx >> 7)) << 4 | (lidx & 127) << 8 : 0u) |
             ((wave == 3 ? gw < 128 - NAGG : (gw >= 7 * W2 || wave == 0 || wave >= 4 + NPW)) ? 4u : 0u);
    }
    role = sreg(role);
  }
  const bool late = role & 1, conv = role & 2, idle = role & 4;
  const unsigned c = (role >> 4) & 7, g = (role >> 8) & 127, wc = role >> 16, pw4 = wave - 4;
  const unsigned l31 = lane & 31, kh = lane >> 5;
  const bool isw = g >= 64;
  const unsigned tile = (g & 63) >> 1, hf = g & 1;
  asm volatile("" : "+s"(px), "+s"(pw), "+s"(pout), "+s"(ws), "+s"(rank), "+s"(wsa[0]), "+s"(wsa[1]), "+s"(wsa[2]), "+s"(wsa[3]),
               "+s"(wsa[4]), "+s"(wsa[5]), "+s"(wsa[6]), "+s"(wsa[7]));
  if (idle && !conv) return;

  f32x4 v[16];
  auto load_unit = [&](unsigned cc) {
    if (!isw) {
      const GAS float* src = (const GAS float*)px + (size_t)(tile * 128 + hf * 64 + l31) * KS + cc * 64 + 4 * kh;
#pragma unroll
      for (int t = 0; t < 16; ++t) v[t] = *(const GAS f32x4*)(src + (t >> 3) * 32 * KS + 8 * (t & 7));
      CFENCE();
    } else {
      const unsigned a = 16 * ((l31 >> 2) & 1) + 4 * (l31 >> 3) + (l31 & 3);
      const GAS char* src = (const GAS char*)pw + (size_t)((cc * 64 + 32 * hf) * NN + tile * 128) * 4;
      const unsigned o = (4 * kh * NN + 4 * a) * 4;
#pragma unroll
      for (int q = 0; q < 16; ++q) {
        unsigned oq = o + (8 * (q >> 2) + (q & 3)) * NN * 4u;
        asm volatile("" : "+v"(oq) :: "memory");
        v[q] = *(const GAS f32x4*)(src + (size_t)oq);
      }
      CFENCE();
    }
  };
  if (wave == (is_gemm ? 0u : 2u)) {
    *(LAS u32x4*)((LAS char*)smem + L_CTRL + lane * 16) = u32x4{0u, 0u, 0u, 0u};
    *(LAS u32x4*)((LAS char*)smem + L_CTRL + 1024 + lane * 16) = u32x4{0u, 0u, 0u, 0u};
    if (!is_gemm && gw < 7 * (NF - NPUSH)) lds_w32(lds0 + L_LOCK + 4, 1u);
  }
  if (conv && c >= 2) {
    __builtin_amdgcn_s_barrier();
    __builtin_amdgcn_s_sleep(SLP2);
    if (c >= 4) __builtin_amdgcn_s_sleep(SLP4);
    if (c >= 6) __builtin_amdgcn_s_sleep(SLP3);
  }
  if (conv) load_unit(c);
  if (!(conv && c >= 2)) {
    WAITL();
    __builtin_amdgcn_s_barrier();
  }
  CFENCE();

  GAS unsigned* ctrp = (GAS unsigned*)(ws + OFF_CTR + (wg * 8 + wave) * 4);
  const unsigned ctr = __hip_atomic_load(ctrp, __ATOMIC_RELAXED, __HIP_MEMORY_SCOPE_AGENT);
  char* peer_v = wsa[0];
#pragma unroll
  for (int q = 1; q < NR; ++q) peer_v = (lane & 7) == (unsigned)q ? wsa[q] : peer_v;
  const unsigned vo = lane * 16;
  const unsigned wo = (lane >> 3) * NN + (lane & 7) * 16;
  char* const ft = ws + OFF_TMO;

  unsigned call;
  if (!conv) {
    call = 0;
    if (is_gemm || wave == 2 || wave == 3) {
      call = rfl(ctr) + 1;
      if (lane == 0) st4v((char*)ctrp, call);
    }
  } else {
    const unsigned grp = isw ? (tile >> 3) : (tile >> 4);
    const bool mine = grp == (isw ? (rank & 3) : (rank >> 2));
    const unsigned lt = isw ? (tile & 7) : (tile & 15);
    const unsigned step = rank * 8 + c;
    const unsigned go = lt < NPUSH / 2 ? OFF_INW + (lt * 64 + step) * 16384 + hf * 8192 : OFF_EXPG + (rank * 16 + lt * 2 + hf) * 512;
    char* g0 = lane_ptr(peer_v, grp + 4 * hf) + go;
    char* g1 = lane_ptr(peer_v, grp + 4 - 4 * hf) + go;
    asm volatile("" : "+s"(g0), "+s"(g1));
    float mx = 0.f;
#pragma unroll
    for (int t = 0; t < 4; ++t) {
      mx = __builtin_fmaxf(__builtin_fmaxf(__builtin_fabsf(v[t][0]), __builtin_fabsf(v[t][1])), mx);
      mx = __builtin_fmaxf(__builtin_fmaxf(__builtin_fabsf(v[t][2]), __builtin_fabsf(v[t][3])), mx);
    }
    unsigned mean = mean_top(f2u(mx)) + 45;
    asm volatile("" : "+s"(mean));
    u32x4 f[8];
    if (!isw) {
#pragma unroll
      for (int q = 0; q < 8; ++q) {
        const f32x4 a = v[2 * q], b = v[2 * q + 1];
        f[q] = u32x4{pk_bf16(a[0], a[1]), pk_bf16(a[2], a[3]), pk_bf16(b[0], b[1]), pk_bf16(b[2], b[3])};
      }
    } else {
#pragma unroll
      for (int q = 0; q < 8; ++q) {
        const int o = 8 * (q >> 2), rb = q & 3;
        f[q] = u32x4{pk_bf16(v[o][rb], v[o + 1][rb]), pk_bf16(v[o + 2][rb], v[o + 3][rb]),
                     pk_bf16(v[o + 4][rb], v[o + 5][rb]), pk_bf16(v[o + 6][rb], v[o + 7][rb])};
      }
    }
    call = rfl(ctr) + 1;
    asm volatile("" : "+s"(call));
    const bool push = c == 0 && isw && lt < NPUSH / 2;
    if (push) {
      st16x4(g0, vo, f[0], f[1], f[2], f[3]);
      st16x4(g1, vo, f[0], f[1], f[2], f[3]);
      st16x4(g0 + 4096, vo, f[4], f[5], f[6], f[7]);
      st16x4(g1 + 4096, vo, f[4], f[5], f[6], f[7]);
      WAITV(0);
      if (lane < 8 && (lane & 3) == grp) st4v(peer_v + OFF_LOCW + ((lt * 64 + step) * 2 + hf) * 4, call);
    } else if (c == 0 || (LASTQ && c == 7 && !isw && (lt & 3) != 0)) {
      const unsigned li = encode(f, mean, ws, c * 128 + g, (LAS char*)smem + (c == 0 ? 8192 : 16384 + wave * 8192), lane);
      if (isw) {
        WAITV(0);
        const unsigned nlan = (rfl(li) & 0x100) ? 64 : 32;
        if (lane < nlan) {
          st8(g0, lane * 8, u32x2{li, call});
          st8(g1, lane * 8, u32x2{li, call});
        }
      }
      st4(ws + OFF_EXPI + (c * 128 + g) * IU + 5632, lane * 4, li);
    } else {
      encode2(f, mean, ws, c * 128 + g, lds0 + 16384 + wave * 8192, lane);
    }
    if (c < 2 && !(c == 0 && isw && (push || NF == 16))) {
      WAITV(0);
      const unsigned fi = (isw ? 0 : 16) + lt * 2 + hf;
      if (lane < 8 && lane != rank && (isw ? (lane & 3) : (lane >> 2)) == grp)
        st4v(peer_v + (c == 0 ? OFF_EXPE + (rank * 64 + fi) * 128 : OFF_EXPF + (rank * 128 + fi) * 4), call);
    }
    if (c >= 2 && c <= 5) {
      WAITV(0);
      unsigned arr = 0, tk = 0;
      if (lane == 0)
        asm volatile("ds_add_rtn_u32 %0, %1, %2\n\ts_waitcnt lgkmcnt(0)" : "=v"(arr) : "v"(lds0 + L_MARR), "v"(1u) : "memory");
      if (rfl(arr) == (wc >= 48 && wc < 112 ? 5u : 3u)) {
        if (lane == 0)
          asm volatile("global_atomic_add %0, %1, %2, off" SC "\n\ts_waitcnt vmcnt(0)" : "=v"(tk) : "v"(ws + OFF_MIDC), "v"(1u) : "memory");
        if (rfl(tk) + 1 == call * 96 && lane < 8 && lane != rank) st4v(peer_v + OFF_EXPF + (rank * 128 + 126) * 4, call);
      }
    }
    if (mine && !push) {
      char* o = ws + (isw ? OFF_INW : OFF_INX) + (lt * 64 + step) * 16384 + hf * 8192;
      st16x4(o, vo, f[0], f[1], f[2], f[3]);
      st16x4(o + 4096, vo, f[4], f[5], f[6], f[7]);
    }
    if (lane == 0) st4v((char*)ctrp, call);
    WAITV(0);
    if (mine && lane == 0) st4v(ws + (isw ? OFF_LOCW : OFF_LOCX) + ((lt * 64 + step) * 2 + hf) * 4, call);
    if (late) {
      unsigned tk = 0;
      if (lane == 0)
        asm volatile("global_atomic_add %0, %1, %2, off" SC "\n\ts_waitcnt vmcnt(0)" : "=v"(tk) : "v"(ws + OFF_ALLC), "v"(1u) : "memory");
      if ((rfl(tk) & 255) == 255 && lane < 8 && lane != rank) st4v(peer_v + OFF_EXPF + (rank * 128 + 127) * 4, call);
    }
    if (idle) return;
  }

  if (is_gemm) {
    WAITL();
    __builtin_amdgcn_s_barrier();
    CFENCE();
    const unsigned xcd = wg & 7, idx = wg >> 3;
    const unsigned tau = 4 * (xcd >> 1) + (idx >> 2), uu = 4 * (xcd & 1) + (idx & 3);
    const unsigned tt = ((tau & 3) << 2) | (tau >> 2);
    const unsigned z = tau < 4 ? uu * 4 + tau : tau * 8 + uu;

    if (wave < 4) {
      f32x16 acc[4];
#pragma unroll
      for (int rb = 0; rb < 4; ++rb)
#pragma unroll
        for (int r = 0; r < 16; ++r) acc[rb][r] = 0.f;
      {
        unsigned s4, s5, s6, s7, s8, s9, s10, v11, v12;
        asm volatile(CONSUME_ASM
                     : "+v"(acc[0]), "+v"(acc[1]), "+v"(acc[2]), "+v"(acc[3]), "=&s"(s4), "=&s"(s5), "=&s"(s6), "=&s"(s7), "=&s"(s8),
                       "=&s"(s9), "=&s"(s10), "=&v"(v11), "=&v"(v12)
                     : "v"(vo + wave * 4096), "v"(lds0 + vo), "v"(lds0 + L_CF + lane * 8), "v"(lds0 + L_REL + wave),
                       "s"(ws + OFF_INX + tt * 64 * 16384), "v"(1u)
                     : "memory", "scc", CONSUME_CLOB);
      }
      float mf[4];
#pragma unroll
      for (int rb = 0; rb < 4; ++rb) {
        mf[rb] = 0.f;
#pragma unroll
        for (int r = 0; r < 16; r += 2)
          mf[rb] = __builtin_fmaxf(__builtin_fmaxf(__builtin_fabsf(acc[rb][r]), __builtin_fabsf(acc[rb][r + 1])), mf[rb]);
      }
      const unsigned ml = umax(umax(f2u(mf[0]), f2u(mf[1])), umax(f2u(mf[2]), f2u(mf[3])));
      const unsigned m = wave_max(ml);
      {
        unsigned arr = 0;
        if (lane == 0)
          asm volatile("ds_max_u32 %1, %2\n\tds_add_rtn_u32 %0, %1, %3 offset:4\n\ts_waitcnt lgkmcnt(0)"
                       : "=v"(arr) : "v"(lds0 + L_WMAX), "v"(m), "v"(1u) : "memory");
        if (rfl(arr) == 3) {
          unsigned wm, fail;
          asm volatile("ds_read_b32 %0, %2\n\tds_read_b32 %1, %2 offset:32\n\ts_waitcnt lgkmcnt(0)"
                       : "=&v"(wm), "=&v"(fail) : "v"(lds0 + L_WMAX) : "memory");
          if (lane == 0 && !fail) st8v(ws + OFF_AMAX1 + wg * 128, u32x2{wm, call});
        }
      }
      if (z < NCAND) {
        u32x4 f[8];
#pragma unroll
        for (int r = 0; r < 16; ++r)
#pragma unroll
          for (int k = 0; k < 2; ++k) {
            const unsigned d = pk_bf16(acc[2 * k][r], acc[2 * k + 1][r]);
            f[r >> 1][2 * (r & 1) + k] = d;
            acc[2 * k][r] = u2f(d << 16);
            acc[2 * k + 1][r] = u2f(d & 0xffff0000u);
          }
        if ((OLDM >> (z >> 3)) & 1) {
          const unsigned li = encode(f, mean_top(ml), ws, 1024 + z * 4 + wave, (LAS char*)smem + L_ENC + wave * 7168, lane);
          st4(ws + OFF_EXPI + (1024 + z * 4 + wave) * IU + 5632, lane * 4, li);
        } else {
          encode2(f, mean_top(ml), ws, 1024 + z * 4 + wave, lds0 + L_ENC + wave * 7168, lane);
        }
      } else if (z >= ZA && z < ZB) {
        LAS char* sg = (LAS char*)smem + L_ENC + wave * 7168;
#pragma unroll
        for (int cq = 0; cq < 4; ++cq) {
          unsigned u = 0;
#pragma unroll
          for (int i = 15; i >= 0; --i) u = __builtin_amdgcn_alignbit(u, f2u(acc[i >> 2][4 * cq + (i & 3)]), 31);
          *(LAS unsigned short*)(sg + ((l31 & 7) * 8 + 4 * kh + cq) * 8 + (l31 >> 3) * 2) = (unsigned short)u;
        }
        CFENCE();
        u32x2 t = *(const LAS u32x2*)(sg + lane * 8);
#pragma unroll
        for (int h = 0; h < 2; ++h) {
          const unsigned a = perm(t[h], t[h], 0x0c010c00u), b = perm(t[h], t[h], 0x0c030c02u);
          t[h] = bfi(M0F, a | (a << 4), (b << 4) | (b << 8));
        }
        st8(ws + OFF_SGN + ((rank * 128 + z) * 4 + wave) * 512, lane * 8, t);
      }
      if (z < NCAND || (z >= ZA && z < ZB)) {
        WAITV(0);
        unsigned arr = 0, fail = 0;
        if (lane == 0)
          asm volatile("ds_add_rtn_u32 %0, %2, %3 offset:8\n\tds_read_b32 %1, %2\n\ts_waitcnt lgkmcnt(0)"
                       : "=&v"(arr), "=&v"(fail) : "v"(lds0 + L_FAIL), "v"(1u) : "memory");
        if (rfl(arr) == 3) {
          const bool good = rfl(fail) == 0;
          if (z < NCAND) {
            if (good && lane < 8 && lane != rank) st4v(peer_v + OFF_BFF + (rank * 64 + z) * 4, call);
          } else {
            unsigned tk = 0;
            if (lane == 0)
              asm volatile("global_atomic_add %0, %1, %2, off" SC "\n\ts_waitcnt vmcnt(0)"
                           : "=v"(tk) : "v"(ws + OFF_SGC + (z / SGG) * 128), "v"(1u) : "memory");
            if ((rfl(tk) & (SGG - 1)) == SGG - 1 && good && lane < 4 * SGG && (lane & 7) != rank)
              asm volatile("global_store_dwordx4 %0, %1, off" SC "\n\ts_nop 1"
                           :: "v"(peer_v + OFF_SGF + (rank * 64 + (z & ~(SGG - 1u)) / 2 + (lane >> 3)) * 16), "v"(u32x4{call, call, 0u, 0u})
                           : "memory");
          }
        }
      }
      u32x2 g;
      for (;;) {
        g = lds_r64(lds0 + L_GMAX);
        if (rfl(g[1]) != 0) break;
        __builtin_amdgcn_s_sleep(1);
      }
      const bool ok = rfl(g[1]) == 1;
      const float inv = u2f(g[0]);
      u32x4 q[4];
#pragma unroll
      for (int r = 0; r < 16; ++r)
        q[r >> 2][r & 3] = pack4(quant(acc[0][r], inv), quant(acc[1][r], inv), quant(acc[2][r], inv), quant(acc[3][r], inv));
      LAS char* tb = (LAS char*)smem + wave * 4096;
#pragma unroll
      for (int jj = 0; jj < 4; ++jj) *(LAS u32x4*)(tb + l31 * 128 + (((4 * kh + jj) ^ (l31 & 7)) * 16)) = q[jj];
      CFENCE();
#pragma unroll
      for (int jj = 0; jj < 4; ++jj) {
        const unsigned row = 8 * jj + (lane >> 3);
        q[jj] = *(const LAS u32x4*)(tb + row * 128 + (((lane & 7) ^ (row & 7)) * 16));
      }
      unsigned vneed = 1;
      if (!ok) {
        q[0] = q[1] = q[2] = q[3] = u32x4{M80, M80, M80, M80};
      } else if (z >= ZR) {
        st16x4(ws + OFF_EXPQ + (z * 4 + wave) * 4096, vo, q[0], q[1], q[2], q[3]);
      } else if (z >= NSKIP) {
        if (z >= ZA && z < ZB) {
          char* e = ws + OFF_EXPO + (z * 4 + wave) * OU;
          unsigned h[16], lw3[6];
          u32x4 w1;
#pragma unroll
          for (int i = 0; i < 16; ++i) {
            const unsigned d = q[i >> 2][i & 3], sm = smear(d);
            h[i] = (d ^ sm) + (sm & M01);
          }
#pragma unroll
          for (int k = 0; k < 2; ++k) {
            const unsigned* A = h + 8 * k;
            lw3[3 * k + 0] = bfi(M07, A[0], bfi(0x38383838u, A[1] << 3, A[6] << 6));
            lw3[3 * k + 1] = bfi(M07, A[2], bfi(0x38383838u, A[3] << 3, bfi(0x40404040u, A[6] << 4, A[7] << 5)));
            lw3[3 * k + 2] = bfi(M07, A[4], bfi(0x38383838u, A[5] << 3, A[7] << 6));
          }
          st16(e, vo, u32x4{lw3[0], lw3[1], lw3[2], lw3[3]});
          st8(e + 1024, lane * 8, u32x2{lw3[4], lw3[5]});
#pragma unroll
          for (int i = 0; i < 16; ++i) h[i] = (h[i] >> 3) & M0F;
          const unsigned tsc = lds0 + 65536 + wave * 8192;
          const unsigned hdr = tenc<false>(h, 0x03030303u, 0x03020100u, 0x03030201u, 0x00030303u, 3u, 0u, tsc, lane, w1);
          if (hdr >> 31) {
            st16x4(ws + OFF_EXPQ + (z * 4 + wave) * 4096, vo, q[0], q[1], q[2], q[3]);
            if (lane == 0) st4(e + 3072, 0u, hdr);
          } else {
            st16(e + 2048, vo, w1);
            const u32x4 v = *(const LAS u32x4*)(size_t)(tsc + 6144 + vo);
            vneed = tlines(hdr);
            if (lane < 8 * vneed) st16(e + 3072, vo, v);
          }
        } else {
          unsigned pk[14], fm[2];
#pragma unroll
          for (int h = 0; h < 2; ++h) {
            const unsigned d7 = q[2 * h + 1][3];
            unsigned fl = 0;
#pragma unroll
            for (int i = 0; i < 8; ++i) {
              const unsigned d = q[2 * h + (i >> 2)][i & 3];
              fl |= ((d ^ (d << 1)) >> (7 - i)) & (M80 >> (7 - i));
              if (i < 7) pk[7 * h + i] = bfi(M80, d7 << (7 - i), d);
            }
            fm[h] = fl;
          }
          char* e = ws + OFF_EXPO + (z * 4 + wave) * OU;
          st16(e, vo, u32x4{pk[0], pk[1], pk[2], pk[3]});
          st16(e + 1024, vo, u32x4{pk[4], pk[5], pk[6], pk[7]});
          st16(e + 2048, vo, u32x4{pk[8], pk[9], pk[10], pk[11]});
          st8(e + 3072, lane * 8, u32x2{pk[12], pk[13]});
          const unsigned cnt = __builtin_popcount(fm[0]) + __builtin_popcount(fm[1]);
          const unsigned inc = wave_scan(cnt);
          const unsigned tot = rdl(inc, 63);
          LAS char* li = (LAS char*)smem + 32768 + wave * 4096;
          if (tot <= 63) {
            unsigned slot = inc - cnt + 1;
            u64 mm = ((u64)fm[1] << 32) | fm[0];
            while (mm) {
              *(LAS unsigned short*)(li + slot * 2) = (unsigned short)(lane * 64 + __builtin_ctzll(mm));
              mm &= mm - 1;
              ++slot;
            }
          } else {
            st8(e + 3712, lane * 8, u32x2{fm[0], fm[1]});
          }
          CFENCE();
          unsigned codes = *(const LAS unsigned*)(li + l31 * 4);
          if (lane == 0) codes = (codes & 0xffff0000u) | (tot <= 63 ? tot : 64u);
          if (lane < 32) st4(e + 3584, lane * 4, codes);
        }
      }
      if (z >= NSKIP) {
        WAITV(0);
        unsigned arr = 0;
        const unsigned mine = 1 + (vneed << (8 + 4 * wave));
        if (lane == 0)
          asm volatile("ds_add_rtn_u32 %0, %1, %2\n\ts_waitcnt lgkmcnt(0)" : "=v"(arr) : "v"(lds0 + L_OARR), "v"(mine) : "memory");
        arr = rfl(arr) + mine;
        if ((arr & 255) == 4) {
          const u32x2 gr = {call, arr >> 8};
          if (z < ZG) {
            if (ok && lane < 8 && lane != rank) st8v(peer_v + OFF_OUTF + (rank * 128 + z) * 8, gr);
          } else {
            if (ok && lane == 0) st8v(ws + OFF_OUTF + (rank * 128 + z) * 8, gr);
            WAITV(0);
            unsigned tk = 0;
            if (lane == 0)
              asm volatile("global_atomic_add %0, %1, %2, off" SC "\n\ts_waitcnt vmcnt(0)"
                           : "=v"(tk) : "v"(ws + OFF_OUTC + (z >> 4) * 128), "v"(1u) : "memory");
            if ((rfl(tk) & 15) == 15 && (lane & 7) != rank) {
              const unsigned fo = OFF_OUTF + (rank * 128 + (z & ~15u)) * 8 + (lane >> 3) * 16;
              u32x4 fl;
              asm volatile("global_load_dwordx4 %0, %1, off" SC "\n\ts_waitcnt vmcnt(0)\n\t"
                           "global_store_dwordx4 %2, %0, off" SC "\n\ts_nop 1"
                           : "=&v"(fl) : "v"(ws + fo), "v"(peer_v + fo) : "memory");
            }
          }
        }
      }
      st16r4(pout + (((rank >> 2) * 2048 + tt * 128 + wave * 32) * NN + (rank & 3) * 1024 + uu * 128),
             wo, wo + 8 * NN, wo + 16 * NN, wo + 24 * NN, q[0], q[1], q[2], q[3]);
    } else if (wave == 4) {
      unsigned src = rank;
      {
        unsigned key[7];
#pragma unroll
        for (int l = 0; l < 7; ++l)
          key[l] = (tau < 4 ? (tau + 8 - l) & 3 : (tau + 8 - (l * 12) / 7) % 12) * 8 + l;
#pragma unroll
        for (int l = 0; l < 7; ++l) {
          unsigned pos = 1;
#pragma unroll
          for (int o = 0; o < 7; ++o) pos += key[o] < key[l] ? 1 : 0;
          if ((lane & 7) == pos) src = (rank + 1 + l) & 7;
        }
      }
      const unsigned stp = src * 8 + (lane >> 3);
      char* fx = ws + OFF_LOCX + (tt * 64 + stp) * 8;
      char* fw = ws + OFF_LOCW + (uu * 64 + stp) * 8;
      u64 loaded = 0, pub = 0, pubk = 0, seen = 0;
      unsigned nl = 0, np = 0, kidx = 0;
      const u64 t0 = rt();
      while (np < 64) {
        u32x2 a, b;
        unsigned tmo;
        asm volatile("global_load_dwordx2 %0, %3, off" SC "\n\tglobal_load_dwordx2 %1, %4, off" SC "\n\t"
                     "global_load_dword %2, %5, off sc1\n\ts_waitcnt vmcnt(0)"
                     : "=&v"(a), "=&v"(b), "=&v"(tmo) : "v"(fx), "v"(fw), "v"(ft) : "memory");
        const u64 wr = __ballot(b[0] >= call && b[1] >= call), xr = __ballot(a[0] >= call && a[1] >= call);
        u64 full = xr & wr;
        if (full == seen && give_up(ws, rfl(tmo), call, t0)) {
          lds_w32(lds0 + L_FAIL, 1u);
          full = ~0ull;
        }
        seen = full;
        const u64 m1 = full & ~loaded;
        if ((m1 >> lane) & 1) {
          kidx = nl + rank_in(m1);
          lds_w32(lds0 + L_LORD + kidx * 4, 0x100u | stp);
        }
        nl += __builtin_popcountll(m1);
        for (;;) {
          const unsigned lim = (pubk == ~0ull ? 64u : (unsigned)__builtin_ctzll(~pubk)) + 8;
          u64 m3 = __ballot(kidx < lim) & full & ~pub;
          if (m3 == 0) break;
          if ((m3 >> lane) & 1) {
            const unsigned i = np + rank_in(m3);
            lds_w32(lds0 + L_CF + i * 8, ((i + 1) << 16) | (kidx << 8) | stp);
          }
          np += __builtin_popcountll(m3);
          pub |= m3;
          for (; m3; m3 &= m3 - 1) pubk |= 1ull << rdl(kidx, __builtin_ctzll(m3));
        }
        const u64 m2 = nl >= PREF ? wr & ~(loaded | m1) & ((xr << 8) | 0xffull) : 0ull;
        if ((m2 >> lane) & 1) {
          kidx = nl + rank_in(m2);
          lds_w32(lds0 + L_LORD + kidx * 4, 0x100u | stp);
        }
        nl += __builtin_popcountll(m2);
        loaded |= m1 | m2;
        while (BACKL && np < 64 && np > BACKL + __builtin_popcountll(__ballot(lds_r32(lds0 + L_REL + lane * 4) == 0x01010101u)))
          __builtin_amdgcn_s_sleep(2);
      }
      for (;;) {
        if (rfl(lds_r32(lds0 + L_WMAX + 4)) >= 4) break;
        __builtin_amdgcn_s_sleep(1);
      }
      float inv;
      const unsigned state = wait_amax<0>(ws, wg, lane, call, inv);
      lds_w64(lds0 + L_GMAX, u32x2{f2u(inv), state});
    } else {
      char* bw = ws + OFF_INW + uu * 64 * 16384;
      const unsigned vo1 = vo + 4096, vo2 = vo + 8192, vo3 = vo + 12288;
      for (unsigned k = wave - 5; k < 64; k += 3) {
        unsigned e;
        for (;;) {
          unsigned rel, lo;
          asm volatile("ds_read_b32 %0, %2\n\tds_read_b32 %1, %3\n\ts_waitcnt lgkmcnt(0)"
                       : "=&v"(rel), "=&v"(lo) : "v"(lds0 + L_REL + ((k - 8) & 63) * 4), "v"(lds0 + L_LORD + k * 4) : "memory");
          e = rfl(lo);
          if (e != 0 && (k < 8 || rfl(rel) == 0x01010101u)) break;
          __builtin_amdgcn_s_sleep(LSLP);
        }
        unsigned keep;
#define DMA4(n) "global_load_lds_dwordx4 %" #n ", %5" SC "\n\tglobal_load_lds_dwordx4 %" #n ", %5 offset:1024" SC "\n\t" \
                "global_load_lds_dwordx4 %" #n ", %5 offset:2048" SC "\n\tglobal_load_lds_dwordx4 %" #n ", %5 offset:3072" SC "\n\t"
        asm volatile("s_nop 4\n\ts_mov_b32 %0, m0\n\ts_mov_b32 m0, %6\n\ts_nop 0\n\t" DMA4(1)
                     "s_add_u32 m0, m0, 0x1000\n\ts_nop 0\n\t" DMA4(2)
                     "s_add_u32 m0, m0, 0x1000\n\ts_nop 0\n\t" DMA4(3)
                     "s_add_u32 m0, m0, 0x1000\n\ts_nop 0\n\t" DMA4(4)
                     "s_mov_b32 m0, %0\n\ts_waitcnt vmcnt(0)"
                     : "=&s"(keep) : "v"(vo), "v"(vo1), "v"(vo2), "v"(vo3), "s"(bw + (e & 63) * 16384), "s"(lds0 + (k & 7) * 16384)
                     : "memory", "scc");
        lds_w32(lds0 + L_CF + k * 8 + 4, 1u);
      }
    }
  } else if (wave == 3) {
    const u64 t0 = rt();
    char* g0 = ws + OFF_AMAX1 + lane * 128;
    for (;;) {
      u32x2 a, b;
      unsigned tmo;
      asm volatile("global_load_dwordx2 %0, %3, off" SC "\n\tglobal_load_dwordx2 %1, %4, off" SC "\n\t"
                   "global_load_dword %2, %5, off sc1\n\ts_waitcnt vmcnt(0)"
                   : "=&v"(a), "=&v"(b), "=&v"(tmo) : "v"(g0), "v"(g0 + 8192), "v"(ft) : "memory");
      const u64 ra = __ballot(a[1] >= call), rb = __ballot(b[1] >= call);
      if ((ra & rb) == ~0ull) {
        const unsigned m = wave_max(umax(a[0], b[0]));
        if (lane < 8 * NREP) st8v(peer_v + OFF_AMAX2 + ((lane >> 3) * 8 + rank) * 128, u32x2{m, call});
        break;
      }
      if (give_up(ws, rfl(tmo), call, t0)) break;
      if ((ra | rb) == 0) __builtin_amdgcn_s_sleep(8);
    }
  } else if (wave == 2) {
    unsigned done = 0, state = 0;
    float inv = 0.f;
    for (;;) {
      const unsigned sl = done % NFS;
      const u32x2 cf = lds_r64(lds0 + L_FCNT);
      const u32x2 de = lds_r64(lds0 + L_FDESC + sl * 8);
      if (rfl(de[1]) != done + 1) {
        if (rfl(cf[1]) == NPW && rfl(cf[0]) == done) break;
        __builtin_amdgcn_s_sleep(16);
        continue;
      }
      if (state == 0) state = wait_amax<FASL>(ws, wg, lane, call, inv);
      const LAS char* tb = (const LAS char*)smem + 16384 + sl * 8192;
      const unsigned od = rfl(de[0]);
      u32x4 q[4];
#pragma unroll
      for (int jj = 0; jj < 4; ++jj) {
        const unsigned row = 8 * jj + (lane >> 3);
#pragma unroll
        for (int h = 0; h < 2; ++h) {
          const u32x4 y = *(const LAS u32x4*)(tb + row * 256 + (((2 * (lane & 7) + h) ^ (row & 7)) * 16));
#pragma unroll
          for (int k = 0; k < 2; ++k)
            q[jj][2 * h + k] = pack4(quant(u2f(y[2 * k] << 16), inv), quant(u2f(y[2 * k] & 0xffff0000u), inv),
                                     quant(u2f(y[2 * k + 1] << 16), inv), quant(u2f(y[2 * k + 1] & 0xffff0000u), inv));
        }
      }
      if (state != 1) q[0] = q[1] = q[2] = q[3] = u32x4{M80, M80, M80, M80};
      st16r4(pout + od, wo, wo + 8 * NN, wo + 16 * NN, wo + 24 * NN, q[0], q[1], q[2], q[3]);
      ++done;
      lds_w32(lds0 + L_FDONE, done);
    }
  } else {
    const unsigned link = gw % 7, slot = gw / 7;
    const unsigned p = (rank + 1 + link) & 7;
    char* const pws = lane_ptr(peer_v, p);
    LAS char* sc = (LAS char*)smem + L_ENC + pw4 * 5120;
#pragma unroll
    for (int q = 0; q < 4; ++q) *(LAS u32x4*)(sc + q * 1024 + vo) = u32x4{0u, 0u, 0u, 0u};
    *(LAS u32x2*)(sc + 4608 + lane * 8) = u32x2{0u, 0u};
    const unsigned n1 = NPUSH + slot;
    const unsigned u1 = (n1 < 8 ? ((n1 >> 1) & 2) + (((n1 >> 1) + link) & 1) : 4 + (((n1 >> 1) + link) & 3)) * 2 + (n1 & 1);
    if (pw4 == 0 && n1 < NF) {
      lds_w32(lds0 + L_DESC + n1 * 4, (64 + (rank & 3) * 16 + u1) | (2048 + (u1 >> 1) * 128 + p * 16 + (u1 & 1)) << 11 | u1 << 24);
    } else {
      unsigned ord, tmp;
#define ORD1(l, v) "s_mov_b32 %1, " #v "\n\tv_writelane_b32 %0, %1, " #l "\n\t"
      asm volatile(ORDER_TABLE(ORD1) "s_nop 0" : "=&v"(ord), "=&s"(tmp));
      const unsigned rots = (link & 1) * 0x1100101u | (link & 3) * 0x1010u | ((link * 12) / 7) << 16;
#pragma unroll 1
      for (unsigned q = 0; q < 3; ++q) {
        const unsigned c = ord & 7, k4 = (ord >> 1) & 28, j = (ord >> 6) & 15;
        ord >>= 10;
        const unsigned xm = 0u - ((0x10010u >> k4) & 1);
        const unsigned jr = j + ((rots >> k4) & 15), jw = jr - ((0x22c4242u >> k4) & 15);
        const unsigned i = (jr < jw ? jr : jw) + ((0x6444200u >> k4) & 15);
        const unsigned lt = bfi(xm, ((i & 3) << 2) | (i >> 2), i), f1 = lt + (xm & 8);
        const unsigned su = c * 128 + lt * 2 + bfi(xm, (rank >> 2) * 32, 64 + (rank & 3) * 16);
        const unsigned d = (~xm & 2048) + (lt * 64 + p * 8 + c) * 2;
        const unsigned c0 = 0u - ((1u >> c) & 1), c1 = 0u - ((2u >> c) & 1);
        const unsigned fi = bfi(c0, 4 * f1, bfi(c1, 2 * f1 + ((f1 & 16) << 2), 112 + ((c + 2) >> 3)));
        const unsigned d0 = su | (d << 11) | (fi << 23);
        *(LAS u32x2*)((LAS char*)smem + L_DESC + (q * 64 + lane) * 8) = u32x2{d0, d0 + 0x801u + (((c0 & 2) | (c1 & 1)) << 23)};
      }
      lds_w32(lds0 + L_TBL, 1u);
    }
    CFENCE();
    const unsigned lw = (4 + ((((lane - 47) >> 1) + link) & 3)) * 2 + ((lane - 47) & 1);
    unsigned efo = OFF_EXPF + p * 512 + (lane & 15) * 8 + (lane >> 5) * 128;
    efo = lane == 56 ? OFF_EXPF + p * 512 + 504 : efo;
    efo = lane - 16 < 32 ? OFF_EXPE + (p * 64 + lane) * 128 : efo;
    efo = lane >= 57 ? OFF_EXPE + (p * 64 + lw) * 128 : efo;
    efo = lane == 63 ? OFF_TMO : efo;
    char* const ef = ws + efo;
    char* const eo = ws + OFF_OUTF + p * 1024 + lane * 16;
    char* const eb = ws + (lane < NCAND ? OFF_BFF + p * 256 + lane * 4 : OFF_TMO);
    char* const es = ws + OFF_SGF + p * 1024 + lane * 16;
    char* const tkp = ws + OFF_TK + link * 128;
    call = rfl(ctr) + 1;
    if (lane == 0) st4v((char*)ctrp, call);
    const unsigned tbase = (call - 1) * (NU - NPUSH + NPW * (W2 - W1)) - NPW * W1 - NPUSH;
    const u64 t0 = rt();
    u64 r0 = 0, r1 = 0, q0 = 0, q1 = 0, qb = 0, qs = 0;
    bool failed = false, parks = true;
    unsigned vln = 3, qinfo = 0;
    unsigned nn = n1 + pw4 * W1;
    char* const e1 = ws + OFF_EXPG + (p * 16 + u1) * 512 + lane * 8;
    unsigned en0 = 0;
    if (slot >= W1) {
      unsigned tk;
      for (unsigned st = 0; st < 2;) {
        unsigned g;
        asm volatile("global_load_dword %0, %1, off" SC "\n\ts_waitcnt vmcnt(0)"
                     : "=&v"(g) : "v"(ws + (lane == 63 ? OFF_TMO : st == 0 ? OFF_GO + link * 128 + (slot >= WM ? 1024 : 0) : OFF_ALLC)) : "memory");
        if (rfl(g) == (st == 0 ? call : call * 256)) { ++st; continue; }
        if (give_up(ws, rdl(g, 63), call, t0)) { failed = true; break; }
        __builtin_amdgcn_s_sleep(XSLP);
      }
      asm volatile("s_mov_b64 exec, 1\n\tglobal_atomic_add %0, %1, %2, off" SC "\n\ts_mov_b64 exec, -1\n\ts_waitcnt vmcnt(0)"
                   : "=&v"(tk) : "v"(tkp), "v"(1u) : "memory");
      nn = rfl(tk) - tbase;
    }
    if (slot < W1 && pw4 >= 2) __builtin_amdgcn_s_sleep(JS2);
    if (slot < W1 && pw4 >= 3) __builtin_amdgcn_s_sleep(JS2);
    while (!failed && nn < NU) {
      if ((nn == XPOS || nn == XPOS2) && lane == 0) st4v(ws + OFF_GO + link * 128 + (nn == XPOS2 ? 1024 : 0), call);
      if (parks && nn >= N1 + 4 * NCAND + NSG) {
        parks = false;
        if (lane == 0) asm volatile("ds_add_u32 %0, %1" :: "v"(lds0 + L_FCNT + 4), "v"(1u) : "memory");
      }
      unsigned csu, cdst, cflag = 0, fi;
      const bool isin = nn < N1, issg = nn - (N1 + 4 * ZS) < NSG;
      const unsigned m = nn - N1 - (nn >= N1 + 4 * ZS ? NSG : 0);
      if (isin) {
        const unsigned de = rfl(lds_r32(lds0 + L_DESC + nn * 4));
        const unsigned d = (de >> 11) & 4095;
        csu = de & 2047;
        cdst = OFF_INX + d * 8192;
        cflag = OFF_LOCX + d * 4;
        fi = nn < 16 ? 2 * (47 + nn) : (de >> 23) & 127;
      } else {
        csu = 1024 + m;
        cdst = out_offset(p, m);
        fi = (m >> 2) & 127;
        if (issg) {
          fi = ZA / 2 + nn - (N1 + 4 * ZS);
          cdst = OFF_SGN + (p * 64 + fi) * 4096;
          cflag = OFF_SGF + (p * 64 + fi) * 16 + 8;
        }
      }
      auto kind = [&]() -> unsigned {
        if (isin) return bit128(r0, r1, fi) ? 1u : 0u;
        if (issg) return ((qs >> fi) & 1) ? 4u : 0u;
        if (bit128(q0, q1, fi)) return fi >= ZR ? 5u : 2u;
        return (fi < NCAND && ((qb >> fi) & 1)) ? 3u : 0u;
      };
      unsigned ck = rfl(kind());
      while (ck == 0) {
        unsigned tmo;
        if (nn < NF) {
          u32x2 g;
          unsigned b;
          asm volatile("global_load_dwordx2 %0, %2, off" SC "\n\tglobal_load_dword %1, %3, off sc1\n\ts_waitcnt vmcnt(0)"
                       : "=&v"(g), "=&v"(b) : "v"(e1), "v"(ft) : "memory");
          const bool on = g[1] >= call;
          en0 = on ? g[0] : 0u;
          tmo = rfl(b);
          if ((__ballot(on) | ((rfl(g[0]) & 0x100) ? 0ull : 0xffffffff00000000ull)) == ~0ull) { ck = 1; break; }
        } else if (isin) {
          poll128(ef, call, r0, r1, tmo);
        } else {
          u32x4 g, s4;
          unsigned b;
          asm volatile("global_load_dwordx4 %0, %3, off" SC "\n\tglobal_load_dword %1, %4, off" SC "\n\t"
                       "global_load_dwordx4 %2, %5, off" SC "\n\ts_waitcnt vmcnt(0)"
                       : "=&v"(g), "=&v"(b), "=&v"(s4) : "v"(eo), "v"(eb), "v"(es) : "memory");
          const u64 sg = __ballot(s4[2] >= call || lane < ZA / 2 || lane >= ZB / 2);
          q0 = __ballot(g[0] >= call) & sg;
          q1 = __ballot(g[2] >= call) & sg;
          qs = __ballot(s4[0] >= call && s4[1] >= call);
          qinfo = g[1] | (g[3] << 16);
          qb = __ballot(b >= call);
          tmo = rdl(b, 63);
        }
        ck = rfl(kind());
        if (ck) break;
        if (give_up(ws, tmo, call, t0)) { failed = true; break; }
        if (nn >= NF) __builtin_amdgcn_s_sleep(PSLP);
      }
      if (failed) {
        if (nn < NF) lds_w32(lds0 + L_LOCK, 1u);
        break;
      }
      const bool qt = !isin && !issg && fi >= ZA && fi < ZB;
      const unsigned vl = rfl(qt ? (rdl(qinfo, fi >> 1) >> (16 * (fi & 1) + 4 * (m & 3))) & 15 : vln);
      const bool lastq = LASTQ && isin && nn >= 360;
      const bool fmt1 = isin ? nn < 48 || lastq : !qt && ((OLDM >> (fi >> 3)) & 1);
      const u64 lmask = lastq ? ~0ull : fmt1 ? 0xffffffffull : (vl >= 8 ? ~0ull : (1ull << (8 * vl)) - 1);
      const unsigned old = rfl(nn < NF ? 6u : fmt1 ? 5u : 0u);
      char* sa = pws + rfl(issg ? cdst : ck == 5 ? OFF_EXPQ + m * 4096 : OFF_EXPI + csu * IU);
      char* sb = pws + rfl(OFF_EXPO + m * OU);
      u32x4 lo[4], ha, va;
      u32x2 hb, sgn;
      unsigned en, tv, ts, sp;
      ck = rfl(ck);
      asm volatile("s_mov_b32 %[sp], 0\n\ts_cmp_eq_u32 %[od], 6\n\ts_cbranch_scc1 LC%=\n\t"
                   "s_mov_b64 exec, 1\n\tds_add_rtn_u32 %[tv], %[lk], %[one] offset:4\n\ts_mov_b64 exec, -1\n\t"
                   "s_waitcnt lgkmcnt(0)\n\tv_readfirstlane_b32 %[sp], %[tv]\n\t"
                   "s_cmp_lg_u32 %[k], %[k3]\n\ts_cbranch_scc1 LB%=\n"
                   "LA%=:\n\t"
                   "ds_read_b32 %[tv], %[lk]\n\ts_waitcnt lgkmcnt(0)\n\tv_readfirstlane_b32 %[ts], %[tv]\n\t"
                   "s_sub_u32 %[ts], %[sp], %[ts]\n\ts_cmp_gt_u32 %[ts], 1\n\ts_cbranch_scc1 LA%=\n\t"
                   "s_cmp_eq_u32 %[ts], 0\n\ts_cbranch_scc1 LC%=\n\t"
                   "s_sleep " FSLP "\n\t"
                   "global_load_dword %[tv], %[fo], %[ws]" SC "\n\ts_waitcnt vmcnt(0)\n\t"
                   "v_readfirstlane_b32 %[ts], %[tv]\n\ts_cmp_ge_u32 %[ts], %[call]\n\ts_cselect_b32 %[k], 2, 3\n"
                   "LB%=:\n\t"
                   "ds_read_b32 %[tv], %[lk]\n\ts_waitcnt lgkmcnt(0)\n\tv_readfirstlane_b32 %[ts], %[tv]\n\t"
                   "s_cmp_lg_u32 %[ts], %[sp]\n\ts_cbranch_scc1 LB%=\n"
                   "LC%=:\n\t"
                   "v_add_u32 %[tv], %[sp], %[one]\n\t"
                   "s_cmp_eq_u32 %[k], 2\n\ts_cbranch_scc1 LI2%=\n\t"
                   "global_load_dwordx4 %[l0], %[vo], %[sa]" SC "\n\tglobal_load_dwordx4 %[l1], %[vo], %[sa] offset:1024" SC "\n\t"
                   "global_load_dwordx4 %[l2], %[vo], %[sa] offset:2048" SC "\n\tglobal_load_dwordx4 %[l3], %[vo], %[sa] offset:3072" SC "\n\t"
                   "s_cmp_ge_u32 %[k], 4\n\ts_cbranch_scc1 LI1%=\n\t"
                   "global_load_dwordx4 %[ha], %[oh], %[sa]" SC "\n\t"
                   "s_cmp_lg_u32 %[od], 0\n\ts_cbranch_scc1 LI3%=\n\t"
                   "s_mov_b64 exec, %[lm]\n\tglobal_load_dwordx4 %[va], %[oh], %[sa] offset:1024" SC "\n\t"
                   "s_mov_b64 exec, -1\n\ts_waitcnt vmcnt(" RELA ")\n\ts_branch LI5%=\n"
                   "LI3%=:\n\t"
                   "global_load_dwordx2 %[hb], %[o8], %[sa] offset:1024" SC "\n\t"
                   "v_mov_b32 %[en], 0\n\ts_mov_b64 exec, %[lm]\n\tglobal_load_dword %[en], %[o4], %[sa] offset:1536" SC "\n\t"
                   "s_mov_b64 exec, -1\n\ts_branch LI1%=\n"
                   "LI2%=:\n\t"
                   "global_load_dwordx4 %[l0], %[vo], %[sb]" SC "\n\tglobal_load_dwordx4 %[l2], %[vo], %[sb] offset:2048" SC "\n\t"
                   "s_cmp_eq_u32 %[qt], 0\n\ts_cbranch_scc1 LI4%=\n\t"
                   "global_load_dwordx2 %[hb], %[o8], %[sb] offset:-3072" SC "\n\t"
                   "s_mov_b64 exec, %[lm]\n\tglobal_load_dwordx4 %[va], %[vo], %[sb] offset:3072" SC "\n\t"
                   "s_mov_b64 exec, -1\n\tglobal_load_dwordx2 %[sg], %[so], %[ws]" SC "\n\t"
                   "s_waitcnt vmcnt(" RELQ ")\n\ts_branch LI5%=\n"
                   "LI4%=:\n\t"
                   "global_load_dwordx4 %[l1], %[vo], %[sb] offset:1024" SC "\n\t"
                   "global_load_dwordx2 %[hb], %[o8], %[sb] offset:-1024" SC "\n\t"
                   "global_load_ushort %[en], %[o2], %[sb] offset:3584" SC "\n"
                   "LI1%=:\n\t"
                   "s_waitcnt vmcnt(" REL ")\n"
                   "LI5%=:\n\tds_write_b32 %[lk], %[tv]\n\ts_waitcnt vmcnt(0)"
                   : [l0] "=&v"(lo[0]), [l1] "=&v"(lo[1]), [l2] "=&v"(lo[2]), [l3] "=&v"(lo[3]), [ha] "=&v"(ha), [hb] "=&v"(hb),
                     [va] "=&v"(va), [sg] "=&v"(sgn), [en] "=&v"(en), [tv] "=&v"(tv), [ts] "=&s"(ts), [sp] "=&s"(sp), [k] "+&s"(ck)
                   : [vo] "v"(vo), [oh] "v"(vo + 4096), [o8] "v"(lane * 8 + 4096), [o4] "v"(lane * 4 + 4096), [o2] "v"(lane * 2),
                     [one] "v"(1u), [lm] "s"(lmask), [od] "s"(old), [k3] "s"(rfl(fi < NSKIP ? 7u : 3u)), [qt] "s"(rfl(qt ? 9u : 0u)), [sa] "s"(sa), [sb] "s"(sb), [lk] "v"(lds0 + L_LOCK),
                     [fo] "v"(OFF_OUTF + p * 1024 + fi * 8), [so] "v"(OFF_SGN + (p * 512 + (m & 511)) * 512 + lane * 8), [ws] "s"(ws), [call] "s"(call)
                   : "memory", "scc");
      if (nn < NF) {
        en = en0;
        while (rfl(lds_r32(lds0 + L_TBL)) == 0) __builtin_amdgcn_s_sleep(1);
      }
      if (issg) {
        st16x4(ws + rfl(cdst), vo, lo[0], lo[1], lo[2], lo[3]);
      } else if (ck == 5) {
        st16r4(pout + rfl(cdst), wo, wo + 8 * NN, wo + 16 * NN, wo + 24 * NN, lo[0], lo[1], lo[2], lo[3]);
      } else if (qt) {
        const unsigned hdr = rfl(va[0]);
        u32x4 q[4];
        if (hdr >> 31) {
          asm volatile("s_nop 4\n\tglobal_load_dwordx4 %0, %4, %5" SC "\n\tglobal_load_dwordx4 %1, %4, %5 offset:1024" SC "\n\t"
                       "global_load_dwordx4 %2, %4, %5 offset:2048" SC "\n\tglobal_load_dwordx4 %3, %4, %5 offset:3072" SC "\n\t"
                       "s_waitcnt vmcnt(0)"
                       : "=&v"(q[0]), "=&v"(q[1]), "=&v"(q[2]), "=&v"(q[3])
                       : "v"(vo), "s"(pws + rfl(OFF_EXPQ + m * 4096)) : "memory");
        } else {
          *(LAS u32x4*)(sc + vo) = va;
          CFENCE();
          unsigned x[16];
          tdec<false>(lo[2], hdr, 0x100800u, 0x282018u, 0x30303030u, (unsigned)(size_t)sc, lane, x);
          const unsigned lw3[6] = {lo[0][0], lo[0][1], lo[0][2], lo[0][3], hb[0], hb[1]};
#pragma unroll
          for (int k = 0; k < 2; ++k) {
            const unsigned p0 = lw3[3 * k], p1 = lw3[3 * k + 1], p2 = lw3[3 * k + 2];
            unsigned* A = x + 8 * k;
            A[0] = bfi(M07, p0, A[0]);
            A[1] = bfi(M07, p0 >> 3, A[1]);
            A[2] = bfi(M07, p1, A[2]);
            A[3] = bfi(M07, p1 >> 3, A[3]);
            A[4] = bfi(M07, p2, A[4]);
            A[5] = bfi(M07, p2 >> 3, A[5]);
            A[6] = bfi(M03, p0 >> 6, bfi(0x04040404u, p1 >> 4, A[6]));
            A[7] = bfi(M03, p2 >> 6, bfi(0x04040404u, p1 >> 5, A[7]));
#pragma unroll
            for (int i = 0; i < 8; ++i) {
              const unsigned e = (sgn[k] >> i) & M01, s8 = e << 7;
              q[2 * k + (i >> 2)][i & 3] = ((A[i] ^ (s8 - e)) + e) ^ s8;
            }
          }
        }
        st16r4(pout + rfl(cdst), wo, wo + 8 * NN, wo + 16 * NN, wo + 24 * NN, q[0], q[1], q[2], q[3]);
      } else if (ck == 2) {
        const unsigned tot = rfl(en);
        LAS char* bm = sc + 4608;
        u32x2 fm;
        if (tot > 63) {
          asm volatile("s_nop 4\n\tglobal_load_dwordx2 %0, %1, %2 offset:3712" SC "\n\ts_waitcnt vmcnt(0)"
                       : "=&v"(fm) : "v"(lane * 8), "s"(pws + rfl(OFF_EXPO + (csu - 1024) * OU)) : "memory");
        } else {
          if (lane >= 1 && lane <= tot)
            asm volatile("ds_or_b32 %0, %1" :: "v"((unsigned)(size_t)bm + ((en >> 5) & 127) * 4), "v"(1u << (en & 31)) : "memory");
          fm = *(const LAS u32x2*)(bm + lane * 8);
          CFENCE();
          *(LAS u32x2*)(bm + lane * 8) = u32x2{0u, 0u};
          CFENCE();
        }
        const unsigned pq[14] = {lo[0][0], lo[0][1], lo[0][2], lo[0][3], lo[1][0], lo[1][1], lo[1][2], lo[1][3],
                                 lo[2][0], lo[2][1], lo[2][2], lo[2][3], hb[0], hb[1]};
        u32x4 q[4];
#pragma unroll
        for (int h = 0; h < 2; ++h) {
          unsigned d7 = 0;
#pragma unroll
          for (int i = 0; i < 7; ++i) {
            const unsigned w = pq[7 * h + i];
            d7 |= (w >> (7 - i)) & (M80 >> (7 - i));
            q[2 * h + (i >> 2)][i & 3] = bfi(M80, (w << 1) ^ (fm[h] << (7 - i)), w);
          }
          q[2 * h + 1][3] = bfi(M80, (d7 << 1) ^ fm[h], d7);
        }
        st16r4(pout + rfl(cdst), wo, wo + 8 * NN, wo + 16 * NN, wo + 24 * NN, q[0], q[1], q[2], q[3]);
      } else {
        u32x4 f[8];
        if (fmt1) {
          if (nn >= 48) {
#pragma unroll
            for (int q = 0; q < 4; ++q) *(LAS u32x4*)(sc + q * 1024 + vo) = u32x4{0u, 0u, 0u, 0u};
            CFENCE();
          }
          const unsigned hdr = rfl(en);
          const unsigned e0 = hdr & 255;
          u32x4 xp[4];
          if (hdr & 0x200) {
            asm volatile("s_nop 4\n\tglobal_load_dwordx4 %0, %4, %5" SC "\n\tglobal_load_dwordx4 %1, %4, %5 offset:1024" SC "\n\t"
                         "global_load_dwordx4 %2, %4, %5 offset:2048" SC "\n\tglobal_load_dwordx4 %3, %4, %5 offset:3072" SC "\n\t"
                         "s_waitcnt vmcnt(0)"
                         : "=&v"(xp[0]), "=&v"(xp[1]), "=&v"(xp[2]), "=&v"(xp[3])
                         : "v"(vo), "s"(pws + rfl(OFF_EXPP + csu * 4096)) : "memory");
          } else {
            if ((hdr & 0x100) && nn >= NF && !lastq) {
              asm volatile("s_nop 4\n\ts_mov_b32 exec_lo, 0\n\tglobal_load_dword %0, %1, %2 offset:1536" SC
                           "\n\ts_mov_b32 exec_lo, -1\n\ts_waitcnt vmcnt(0)"
                           : "+&v"(en) : "v"(lane * 4 + 4096), "s"(pws + rfl(OFF_EXPI + csu * IU)) : "memory");
            }
            if (lane == 0) en &= 0xffff0000u;
            const unsigned x5z = ((256 - e0) & 255) >> 3;
            const unsigned ca = en & 15, cb = (en >> 16) & 15, pa = (en >> 4) & 4095, pb = en >> 20;
            if (ca) *(LAS unsigned char*)(sc + pa) = (unsigned char)((ca == 8 ? x5z : (ca | ((ca & 8) << 1))) << 2);
            if (cb) *(LAS unsigned char*)(sc + pb) = (unsigned char)((cb == 8 ? x5z : (cb | ((cb & 8) << 1))) << 2);
            CFENCE();
#pragma unroll
            for (int q = 0; q < 4; ++q) xp[q] = *(const LAS u32x4*)(sc + q * 1024 + vo);
            CFENCE();
            if (ca) *(LAS unsigned char*)(sc + pa) = 0;
            if (cb) *(LAS unsigned char*)(sc + pb) = 0;
            CFENCE();
          }
          const unsigned base = e0 * 0x00800080u;
          const unsigned hp[6] = {ha[0], ha[1], ha[2], ha[3], hb[0], hb[1]};
#pragma unroll
          for (int h = 0; h < 2; ++h) {
            const unsigned p0 = hp[3 * h], p1 = hp[3 * h + 1], p2 = hp[3 * h + 2];
            unsigned H[8];
            H[0] = bfi(M83, p0, xp[2 * h][0]);
            H[1] = bfi(M83, rotr(p0, 3), xp[2 * h][1]);
            H[2] = bfi(M83, p1, xp[2 * h][2]);
            H[3] = bfi(M83, rotr(p1, 3), xp[2 * h][3]);
            H[4] = bfi(M83, p2, xp[2 * h + 1][0]);
            H[5] = bfi(M83, rotr(p2, 3), xp[2 * h + 1][1]);
            H[6] = bfi(M03, p0 >> 5, bfi(M80, p1 << 2, xp[2 * h + 1][2]));
            H[7] = bfi(M03, p2 >> 5, bfi(M80, p1 << 1, xp[2 * h + 1][3]));
#pragma unroll
            for (int i = 0; i < 8; ++i) {
              const unsigned L = lo[2 * h + (i >> 2)][i & 3];
              f[4 * h + (i >> 1)][2 * (i & 1)] = pk_add(perm(H[i], L, 0x05010400u), base);
              f[4 * h + (i >> 1)][2 * (i & 1) + 1] = pk_add(perm(H[i], L, 0x07030602u), base);
            }
          }
        } else {
          const unsigned hdr = rfl(va[0]), top = hdr & 255;
          unsigned x[16];
          if (hdr >> 31) {
            u32x4 dp[4];
            asm volatile("s_nop 4\n\tglobal_load_dwordx4 %0, %4, %5" SC "\n\tglobal_load_dwordx4 %1, %4, %5 offset:1024" SC "\n\t"
                         "global_load_dwordx4 %2, %4, %5 offset:2048" SC "\n\tglobal_load_dwordx4 %3, %4, %5 offset:3072" SC "\n\t"
                         "s_waitcnt vmcnt(0)"
                         : "=&v"(dp[0]), "=&v"(dp[1]), "=&v"(dp[2]), "=&v"(dp[3])
                         : "v"(vo), "s"(pws + rfl(OFF_EXPP + csu * 4096)) : "memory");
            const unsigned tq = top * M01;
#pragma unroll
            for (int g = 0; g < 16; ++g) {
              const unsigned d = dp[g >> 2][g & 3];
              x[g] = ((tq | M80) - (d & ~M80)) ^ ((tq ^ ~d) & M80);
            }
          } else {
            vln = tlines(hdr);
            if (vln > vl) {
              const u64 more = (vln >= 8 ? ~0ull : (1ull << (8 * vln)) - 1) & ~lmask;
              asm volatile("s_nop 4\n\ts_mov_b64 exec, %3\n\tglobal_load_dwordx4 %0, %1, %2 offset:1024" SC
                           "\n\ts_mov_b64 exec, -1\n\ts_waitcnt vmcnt(0)"
                           : "+&v"(va) : "v"(vo + 4096), "s"(sa), "s"(more) : "memory");
            }
            const unsigned ts2 = (unsigned)(size_t)sc;
            *(LAS u32x4*)(size_t)(ts2 + vo) = va;
            CFENCE();
            tdec<true>(ha, hdr, top * 0x010101u - 0x030201u, top * 0x010101u - 0x050400u, (top - 6) * M01, ts2, lane, x);
          }
#pragma unroll
          for (int g = 0; g < 16; ++g) {
            const unsigned sm = lo[g >> 2][g & 3];
            const unsigned L = bfi(M80, x[g] << 7, sm), H = bfi(M80, sm, x[g] >> 1);
            f[g >> 1][2 * (g & 1)] = perm(H, L, 0x05010400u);
            f[g >> 1][2 * (g & 1) + 1] = perm(H, L, 0x07030602u);
          }
        }
        if (ck == 1) {
          char* o = ws + rfl(cdst);
          st16x4(o, vo, f[0], f[1], f[2], f[3]);
          st16x4(o + 4096, vo, f[4], f[5], f[6], f[7]);
        } else {
          unsigned pi = 0;
          if (lane == 0)
            asm volatile("ds_add_rtn_u32 %0, %1, %2\n\ts_waitcnt lgkmcnt(0)" : "=v"(pi) : "v"(lds0 + L_FCNT), "v"(1u) : "memory");
          pi = rfl(pi);
          while (pi >= NFS && rfl(lds_r32(lds0 + L_FDONE)) + NFS <= pi) __builtin_amdgcn_s_sleep(4);
          const unsigned sl = pi % NFS;
          LAS char* tb = (LAS char*)smem + 16384 + sl * 8192;
#pragma unroll
          for (int q = 0; q < 8; ++q) *(LAS u32x4*)(tb + l31 * 256 + (((8 * kh + q) ^ (l31 & 7)) * 16)) = f[q];
          lds_w64(lds0 + L_FDESC + sl * 8, u32x2{cdst, pi + 1});
        }
      }
      {
        const unsigned rf = rfl(nn < 80 || (isin && !(((r0 & r1) >> 56) & 1)) ? 1u : (nn + 60 >= N1 && ((q0 & q1) | ((1ull << (NSKIP / 2)) - 1)) != ~0ull) ? 2u : 0u);
        u32x2 fg;
        u32x4 f4, s4;
        unsigned fe, tk;
        asm volatile("s_mov_b64 exec, 1\n\tglobal_atomic_add %[tk], %[tkp], %[one], off" SC "\n\ts_mov_b64 exec, -1\n\t"
                     "s_cmp_eq_u32 %[rf], 0\n\ts_cbranch_scc1 LR0%=\n\t"
                     "s_cmp_eq_u32 %[rf], 2\n\ts_cbranch_scc1 LR2%=\n\t"
                     "global_load_dwordx2 %[fg], %[ef], off" SC "\n\ts_branch LR0%=\n"
                     "LR2%=:\n\t"
                     "global_load_dwordx4 %[f4], %[eo], off" SC "\n\tglobal_load_dword %[fe], %[eb], off" SC "\n\t"
                     "global_load_dwordx4 %[s4], %[es], off" SC "\n"
                     "LR0%=:\n\ts_waitcnt vmcnt(0)"
                     : [tk] "=&v"(tk), [fg] "=&v"(fg), [f4] "=&v"(f4), [s4] "=&v"(s4), [fe] "=&v"(fe)
                     : [tkp] "v"(tkp), [one] "v"(1u), [ef] "v"(ef), [eo] "v"(eo), [eb] "v"(eb), [es] "v"(es), [rf] "s"(rf) : "memory", "scc");
        if ((ck == 1 || ck == 4) && lane == 0) st4v(ws + cflag, call);
        if (rf == 1) { r0 = __ballot(fg[0] >= call); r1 = __ballot(fg[1] >= call); }
        if (rf == 2) {
          const u64 sg = __ballot(s4[2] >= call || lane < ZA / 2 || lane >= ZB / 2);
          q0 = __ballot(f4[0] >= call) & sg;
          q1 = __ballot(f4[2] >= call) & sg;
          qs = __ballot(s4[0] >= call && s4[1] >= call);
          qb = __ballot(fe >= call);
          qinfo = f4[1] | (f4[3] << 16);
        }
        nn = rfl(tk) - tbase;
      }
    }
    while (failed && nn < NU) {
      if (nn >= N1 && nn - (N1 + 4 * ZS) >= NSG) {
        const u32x4 bad = {M80, M80, M80, M80};
        st16r4(pout + rfl(out_offset(p, nn - N1 - (nn >= N1 + 4 * ZS ? NSG : 0))),
               wo, wo + 8 * NN, wo + 16 * NN, wo + 24 * NN, bad, bad, bad, bad);
      }
      unsigned t2;
      asm volatile("s_mov_b64 exec, 1\n\tglobal_atomic_add %0, %1, %2, off" SC "\n\ts_mov_b64 exec, -1\n\ts_waitcnt vmcnt(0)"
                   : "=&v"(t2) : "v"(tkp), "v"(1u) : "memory");
      nn = rfl(t2) - tbase;
    }
    if (parks && lane == 0) asm volatile("ds_add_u32 %0, %1" :: "v"(lds0 + L_FCNT + 4), "v"(1u) : "memory");
  }
}

}

extern "C" void kernel_launch(int rank, int nranks, ncclComm_t comm, hipStream_t stream,
                              void* const* d_in, const int* in_sizes, int n_in,
                              void* d_out, int out_size, const int* in_dtypes, int out_dtype,
                              void* const* d_out_all, void* const* d_ws_all, size_t ws_bytes) {
  if (nranks != NR || (unsigned)rank >= (unsigned)NR || ws_bytes < WS_NEED || n_in < 2 || out_size != MM * NN ||
      in_sizes[0] != MM * KS || in_sizes[1] != KS * NN) {
    fprintf(stderr, "kernel_launch: unexpected problem shape\n");
    return;
  }
  hipError_t e = hipFuncSetAttribute((const void*)fused, hipFuncAttributeMaxDynamicSharedMemorySize, (int)LDS_BYTES);
  if (e != hipSuccess) { fprintf(stderr, "hipFuncSetAttribute: %s\n", hipGetErrorString(e)); return; }
  Params P;
  P.x = (const float*)d_in[0];
  P.w = (const float*)d_in[1];
  P.out = (char*)d_out;
  P.ws = (char*)d_ws_all[rank];
  for (int q = 0; q < NR; ++q) P.ws_all[q] = (char*)d_ws_all[q];
  P.rank = rank;
  P.pad = 0;
  hipLaunchKernelGGL(fused, dim3(NWG), dim3(NTHR), LDS_BYTES, stream, P);
  e = hipGetLastError();
  if (e != hipSuccess) fprintf(stderr, "launch: %s\n", hipGetErrorString(e));
}
```
